```python
import numpy as np

kernel_name = "gpu_a2a_gemm_m4096_k4096_n4096_f32_none"


ABSMAX_RTOL = 0.001

COMPUTE_DTYPE = "float32"
EPILOGUE = "none"
IN_QUANT = None
OUT_QUANT = None
IN_DTYPES = {"x": "float32", "w_mat": "float32"}
WORLD_SIZE = 8

def setup_inputs():

    x = np.random.randn(4096, 4096).astype(np.float32)
    w_mat = (np.random.randn(4096, 4096) / np.sqrt(4096)).astype(np.float32)
    return {"x": x, "w_mat": w_mat}

def _epilogue(y):
    return y

def reference(x, w_mat):
    y = (x @ w_mat).astype(np.float32)
    return _epilogue(y)
```

```cpp
#include <hip/hip_runtime.h>
#include <rccl/rccl.h>
#include <cstdio>

#define HIP_CHECK(expr) do { hipError_t _e = (expr); if (_e != hipSuccess) { \
    fprintf(stderr, "HIP error %s at %s:%d\n", hipGetErrorString(_e), __FILE__, __LINE__); \
    return; } } while (0)

typedef unsigned u32x4 __attribute__((ext_vector_type(4)));
typedef float f32x4 __attribute__((ext_vector_type(4)));
typedef _Float16 h2 __attribute__((ext_vector_type(2)));
#define SB() __builtin_amdgcn_sched_barrier(0)

constexpr int NR = 8, MT = 512, N = 4096, K = 4096, KS = 512;
constexpr size_t A_BYTES = (size_t)MT * K * 2;
constexpr size_t OFF_A = 0;
constexpr size_t OFF_FLAG = 2 * A_BYTES;
constexpr size_t OFF_TMO = OFF_FLAG + 4096;
constexpr size_t OFF_CNT = OFF_TMO + 4096;
constexpr size_t WS_NEED = OFF_CNT + 4096;
constexpr unsigned long long DEADLINE = 50000000ull;

struct Params { const float* x; const float* w; float* out; char* ws[NR]; int rank; };

__device__ __forceinline__ unsigned cvtpk(float a, float b) { h2 r; r.x = (_Float16)a; r.y = (_Float16)b; return __builtin_bit_cast(unsigned, r); }

constexpr int DA = 2, DW = 4, NW = 4;

__global__ __launch_bounds__(256, 1) void a2a_gemm(Params p) {
  extern __shared__ __attribute__((aligned(16))) char smem[];
  const int lane = threadIdx.x & 63;
  const int wave = __builtin_amdgcn_readfirstlane(threadIdx.x >> 6);
  const int bid = blockIdx.x;
  char* myws = p.ws[p.rank];
  unsigned* cnt = (unsigned*)(myws + OFF_CNT) + bid;
  const unsigned call = __builtin_amdgcn_readfirstlane(__hip_atomic_load(cnt, __ATOMIC_RELAXED, __HIP_MEMORY_SCOPE_AGENT)) + 1;
  const unsigned par = call & 1;
  {
    const int dst = (bid + p.rank) & 7, rb = bid >> 3;
    const float* xr = p.x + (size_t)(dst * MT + rb * 16 + (lane & 15)) * KS + (lane >> 4) * 8 + wave * 128;
    f32x4 v[8];
#pragma unroll
    for (int s = 0; s < 4; s++) { v[2 * s] = *(const f32x4*)(xr + s * 32); v[2 * s + 1] = *(const f32x4*)(xr + s * 32 + 4); }
    char* dp = p.ws[dst] + OFF_A + par * A_BYTES + ((size_t)((rb >> 3) * 128 + p.rank * 16 + wave * 4) * 8 + (rb & 7)) * 1024 + lane * 16;
#pragma unroll
    for (int s = 0; s < 4; s++) {
      u32x4 o = {cvtpk(v[2 * s][0], v[2 * s][1]), cvtpk(v[2 * s][2], v[2 * s][3]), cvtpk(v[2 * s + 1][0], v[2 * s + 1][1]), cvtpk(v[2 * s + 1][2], v[2 * s + 1][3])};
      asm volatile("global_store_dwordx4 %0, %1, off sc0 sc1\n\ts_nop 1" :: "v"(dp + s * 8192), "v"(o) : "memory");
    }
    asm volatile("s_waitcnt vmcnt(0)" ::: "memory");
    __syncthreads();
    if (threadIdx.x == 0)
      __hip_atomic_fetch_add((unsigned*)(p.ws[dst] + OFF_FLAG + p.rank * 128), 1u, __ATOMIC_RELAXED, __HIP_MEMORY_SCOPE_SYSTEM);
  }
  const int xcd = bid & 7, idx = bid >> 3;
  const int mt = idx & 3, nt = xcd * 8 + (idx >> 2);
  auto ra = __builtin_amdgcn_make_buffer_rsrc((void*)(myws + OFF_A + par * A_BYTES), 0, (int)A_BYTES, 0x00020000);
  auto rw = __builtin_amdgcn_make_buffer_rsrc((void*)p.w, 0, K * N * 4, 0x00020000);
  const int va = lane * 16;
  const int vw = (lane >> 4) * 8 * N * 4 + (lane & 15) * 16;
  constexpr int NS = 128 / NW;
  int sa = (mt * 128 + wave) * 8192;
  int sw = wave * 32 * N * 4 + nt * 256;
  f32x4 acc[8][4];
#pragma unroll
  for (int m = 0; m < 8; m++) for (int j = 0; j < 4; j++) acc[m][j] = f32x4{0, 0, 0, 0};
  u32x4 ab[DA][8]; u32x4 wb[DW][8];
#pragma unroll
  for (int d = 0; d < DW; d++) {
#pragma unroll
    for (int i = 0; i < 8; i++) wb[d][i] = __builtin_amdgcn_raw_buffer_load_b128(rw, vw, sw + i * N * 4, 0);
    sw += NW * 32 * N * 4;
  }
  SB();
  unsigned* tmo = (unsigned*)(myws + OFF_TMO);
  if (threadIdx.x < NR) {
    unsigned* f = (unsigned*)(myws + OFF_FLAG + threadIdx.x * 128);
    unsigned long long t0 = __builtin_amdgcn_s_memrealtime();
    while (__hip_atomic_load(f, __ATOMIC_RELAXED, __HIP_MEMORY_SCOPE_SYSTEM) < call * 32u) {
      if (__hip_atomic_load(tmo, __ATOMIC_RELAXED, __HIP_MEMORY_SCOPE_AGENT) == call || __builtin_amdgcn_s_memrealtime() - t0 > DEADLINE) {
        __hip_atomic_store(tmo, call, __ATOMIC_RELAXED, __HIP_MEMORY_SCOPE_AGENT);
        break;
      }
      __builtin_amdgcn_s_sleep(2);
    }
  }
  __syncthreads();
  const bool bad = __hip_atomic_load(tmo, __ATOMIC_RELAXED, __HIP_MEMORY_SCOPE_AGENT) == call;
#pragma unroll
  for (int d = 0; d < DA; d++) {
#pragma unroll
    for (int m = 0; m < 8; m++) ab[d][m] = __builtin_amdgcn_raw_buffer_load_b128(ra, va + (m & 3) * 1024, sa + (m >> 2) * 4096, 17);
    sa += NW * 8192;
  }
  SB();
  constexpr int UN = 4;
  for (int s = 0; s < NS; s += UN) {
#pragma unroll
    for (int u = 0; u < UN; u++) {
      const int da = u % DA, dw = u % DW;
      u32x4 bf[4];
#pragma unroll
      for (int j = 0; j < 4; j++)
#pragma unroll
        for (int q = 0; q < 4; q++) { f32x4 w0 = __builtin_bit_cast(f32x4, wb[dw][2 * q]), w1 = __builtin_bit_cast(f32x4, wb[dw][2 * q + 1]); bf[j][q] = cvtpk(w0[j], w1[j]); }
      SB();
      const int sw_e = (s + u + DW < NS) ? sw : 0;
#pragma unroll
      for (int i = 0; i < 8; i++) wb[dw][i] = __builtin_amdgcn_raw_buffer_load_b128(rw, vw, sw_e + i * N * 4, 0);
      sw += NW * 32 * N * 4;
      SB();
      const int sa_e = (s + u + DA < NS) ? sa : 0;
#pragma unroll
      for (int m = 0; m < 8; m++) {
#pragma unroll
        for (int j = 0; j < 4; j++) asm volatile("v_mfma_f32_16x16x32_f16 %0, %1, %2, %0" : "+a"(acc[m][j]) : "v"(ab[da][m]), "v"(bf[j]));
        SB();
        ab[da][m] = __builtin_amdgcn_raw_buffer_load_b128(ra, va + (m & 3) * 1024, sa_e + (m >> 2) * 4096, 17);
        SB();
      }
      sa += NW * 8192;
    }
  }
  asm volatile("s_nop 15");
  f32x4* sl = (f32x4*)smem;
#pragma unroll
  for (int m = 0; m < 8; m++) { if ((m >> 1) != wave) {
      int slot = ((m >> 1) > wave) ? wave : wave - 1;
#pragma unroll
      for (int j = 0; j < 4; j++) sl[(((m * 3) + slot) * 4 + j) * 64 + lane] = acc[m][j]; } }
  __syncthreads();
  const float nanv = __builtin_nanf("");
#pragma unroll
  for (int m = 0; m < 8; m++) { if ((m >> 1) == wave) {
      f32x4 r[4];
#pragma unroll
      for (int j = 0; j < 4; j++) { r[j] = acc[m][j];
#pragma unroll
        for (int q = 0; q < 3; q++) r[j] += sl[(((m * 3) + q) * 4 + j) * 64 + lane]; }
#pragma unroll
      for (int g = 0; g < 4; g++) { f32x4 o = {r[0][g], r[1][g], r[2][g], r[3][g]};
        if (bad) o = f32x4{nanv, nanv, nanv, nanv};
        *(f32x4*)(p.out + (size_t)(mt * 128 + m * 16 + (lane >> 4) * 4 + g) * N + nt * 64 + (lane & 15) * 4) = o; }
    } }
  if (threadIdx.x == 0) __hip_atomic_store(cnt, call, __ATOMIC_RELAXED, __HIP_MEMORY_SCOPE_AGENT);
}

extern "C" void kernel_launch(int rank, int nranks, ncclComm_t comm, hipStream_t stream,
                              void* const* d_in, const int* in_sizes, int n_in,
                              void* d_out, int out_size, const int* in_dtypes, int out_dtype,
                              void* const* d_out_all, void* const* d_ws_all, size_t ws_bytes) {
  if (nranks != NR || (unsigned)rank >= (unsigned)NR || ws_bytes < WS_NEED) { fprintf(stderr, "bad config\n"); return; }
  static bool init = false;
  constexpr int LDS = 98304;
  if (!init) { HIP_CHECK(hipFuncSetAttribute((const void*)a2a_gemm, hipFuncAttributeMaxDynamicSharedMemorySize, LDS)); init = true; }
  Params p;
  p.x = (const float*)d_in[0]; p.w = (const float*)d_in[1]; p.out = (float*)d_out; p.rank = rank;
  for (int i = 0; i < NR; i++) p.ws[i] = (char*)d_ws_all[i];
  hipLaunchKernelGGL(a2a_gemm, dim3(256), dim3(256), LDS, stream, p);
  HIP_CHECK(hipGetLastError());
}
```

```cpp
#include <hip/hip_runtime.h>
#include <rccl/rccl.h>
#include <cstdio>

#define HIP_CHECK(expr) do { hipError_t _e = (expr); if (_e != hipSuccess) { \
    fprintf(stderr, "HIP error %s at %s:%d\n", hipGetErrorString(_e), __FILE__, __LINE__); \
    return; } } while (0)

typedef unsigned u32x4 __attribute__((ext_vector_type(4)));
typedef float f32x4 __attribute__((ext_vector_type(4)));
typedef _Float16 h2 __attribute__((ext_vector_type(2)));
#define LAS __attribute__((address_space(3)))
#define GAS __attribute__((address_space(1)))
#define SB() __builtin_amdgcn_sched_barrier(0)
#define MEMBAR() asm volatile("" ::: "memory")

constexpr int NR = 8, MT = 512, N = 4096, K = 4096, KS = 512;
constexpr unsigned A_BYTES = MT * K * 2;
constexpr unsigned OFF_A = 0;
constexpr unsigned OFF_AFLAG = 2 * A_BYTES;
constexpr unsigned OFF_CFLAG = OFF_AFLAG + 4096;
constexpr unsigned OFF_XTAB = OFF_CFLAG + 65536;
constexpr unsigned OFF_TMO = OFF_XTAB + 4096;
constexpr unsigned OFF_CNT = OFF_TMO + 4096;
constexpr unsigned WS_NEED = OFF_CNT + 4096;
constexpr unsigned long long DEADLINE = 50000000ull;

#define RING 19
#define LAG1 4
#define LAG2 9
#define LAG3 14
#define ALOOK 3
#define WD 2
#define SLOT0 150
#define SLOTLEN 232
#define XLEAD 250
#define POLL_FIRST 60
#define POLL_SLEEP 4
#define TOUCH_AT 24
#define ROT 0

#define AREC 131072
#define LPRIO 0
constexpr int LDS_CTL = RING * 8192, LDS_BYTES = LDS_CTL + 256;
enum { C_W0 = 0, C_W1 = 1, C_AST = 2, C_MODE = 3, C_DONE = 8 };

constexpr int lag_of(int v) { return v == 0 ? 0 : v == 1 ? LAG1 : v == 2 ? LAG2 : LAG3; }
constexpr int class_of(int v, int s) {
  int c = 0;
  for (int a = 0; a < 16; a++) {
    const int ka = 7 * (a & 3) + lag_of(a >> 2), kb = 7 * s + lag_of(v);
    if (ka < kb || (ka == kb && a < v * 4 + s)) c++;
  }
  return c;
}
constexpr unsigned long long make_fwd() { unsigned long long r = 0; for (int a = 0; a < 16; a++) r |= (unsigned long long)class_of(a >> 2, a & 3) << (4 * a); return r; }
constexpr unsigned long long make_inv() { unsigned long long r = 0; for (int a = 0; a < 16; a++) r |= (unsigned long long)a << (4 * class_of(a >> 2, a & 3)); return r; }
constexpr unsigned long long CLS_FWD = make_fwd(), CLS_INV = make_inv();

struct Params { const float* x; const float* w; float* out; char* ws[NR]; int rank; };

#define STAMP(i)
#define CSTAMP(i)

__device__ __forceinline__ unsigned cvtpk(float a, float b) { h2 r; r.x = (_Float16)a; r.y = (_Float16)b; return __builtin_bit_cast(unsigned, r); }
__device__ __forceinline__ unsigned cvtpk_pin(float a, float b) { unsigned r; asm volatile("v_cvt_pk_f16_f32 %0, %1, %2" : "=v"(r) : "v"(a), "v"(b)); return r; }
__device__ __forceinline__ unsigned lds_ld(LAS unsigned* p) { return __builtin_amdgcn_readfirstlane(__hip_atomic_load(p, __ATOMIC_RELAXED, __HIP_MEMORY_SCOPE_WORKGROUP)); }
__device__ __forceinline__ void lds_st(LAS unsigned* p, unsigned v) { __hip_atomic_store(p, v, __ATOMIC_RELAXED, __HIP_MEMORY_SCOPE_WORKGROUP); }
__device__ __forceinline__ unsigned lds_wait_ge(LAS unsigned* p, unsigned v) {
  unsigned r;
  while ((r = lds_ld(p)) < v) __builtin_amdgcn_s_sleep(1);
  MEMBAR();
  return r;
}
__device__ __forceinline__ bool expired(GAS unsigned* tmo, unsigned call, unsigned long long t0) {
  const unsigned long long dt = __builtin_amdgcn_s_memrealtime() - t0;
  if (dt < 20000) return false;
  if (__hip_atomic_load(tmo, __ATOMIC_RELAXED, __HIP_MEMORY_SCOPE_AGENT) == call || dt > DEADLINE) {
    __hip_atomic_store(tmo, call, __ATOMIC_RELAXED, __HIP_MEMORY_SCOPE_AGENT);
    return true;
  }
  return false;
}
#define MFMA(acc, a, b) asm volatile("v_mfma_f32_16x16x32_f16 %0, %1, %2, %0" : "+v"(acc) : "v"(a), "v"(b))

template <int STA, int LDA>
__device__ __forceinline__ void chain_hop(f32x4 (&acc)[2][8], int v, __amdgpu_buffer_rsrc_t ro, int vo, __amdgpu_buffer_rsrc_t rc, int fo,
                                          unsigned call, GAS unsigned* tmo, int lane) {
  f32x4 o[16];
  if (v > 0) {
    const unsigned long long t0 = __builtin_amdgcn_s_memrealtime();
    while (__builtin_amdgcn_readfirstlane(__builtin_amdgcn_raw_buffer_load_b32(rc, fo, 0, LDA)) < call * 4 + v) {
      MEMBAR();
      if (expired(tmo, call, t0)) break;
      __builtin_amdgcn_s_sleep(1);
    }
    MEMBAR();
#pragma unroll
    for (int r = 0; r < 16; r++) {
      o[r] = __builtin_bit_cast(f32x4, __builtin_amdgcn_raw_buffer_load_b128(ro, vo + ((r >> 2) & 1) * 256, ((r >> 3) * 16 + (r & 3)) * (N * 4), LDA));
      SB();
    }
  } else {
#pragma unroll
    for (int r = 0; r < 16; r++) o[r] = f32x4{0, 0, 0, 0};
  }
  if (v == 3 && __hip_atomic_load(tmo, __ATOMIC_RELAXED, __HIP_MEMORY_SCOPE_AGENT) == call) {
    const float q = __builtin_nanf("");
#pragma unroll
    for (int r = 0; r < 16; r++) o[r] = f32x4{q, q, q, q};
  }
#pragma unroll
  for (int r = 0; r < 16; r++) {
    const int i = r >> 3, hf = (r >> 2) & 1, e = r & 3;
    o[r] += f32x4{acc[i][hf * 4][e], acc[i][hf * 4 + 1][e], acc[i][hf * 4 + 2][e], acc[i][hf * 4 + 3][e]};
    __builtin_amdgcn_raw_buffer_store_b128(__builtin_bit_cast(u32x4, o[r]), ro, vo + hf * 256, (i * 16 + e) * (N * 4), STA);
    SB();
    asm volatile("s_nop 1");
    SB();
  }
  if (v < 3) {
    asm volatile("s_waitcnt vmcnt(0)" ::: "memory");
    if (lane == 0) __builtin_amdgcn_raw_buffer_store_b32(call * 4 + v + 1, rc, fo, 0, STA);
  }
}

__global__ __launch_bounds__(768, 3) void a2a_gemm(Params p) {
  extern __shared__ __attribute__((aligned(16))) char smem_[];
  LAS char* smem = (LAS char*)smem_;
  LAS unsigned* ctl = (LAS unsigned*)(smem + LDS_CTL);
  const int lane = threadIdx.x & 63;
  const int wave = __builtin_amdgcn_readfirstlane(threadIdx.x >> 6);
  const int bid = blockIdx.x;
  const int rank = p.rank;
  char* myws = p.ws[rank];
  const int idx = bid >> 3;
  const int kq = idx & 3, mh = (idx >> 2) & 1, nt = (bid & 7) * 4 + (idx >> 3);
  GAS unsigned* cnt = (GAS unsigned*)(myws + OFF_CNT) + bid;
  GAS unsigned* tmo = (GAS unsigned*)(myws + OFF_TMO);
  if (wave == 0) STAMP(0);

  if (wave < 8) {
    if (wave == 0 && lane < 32) lds_st(ctl + lane, 0);
    const unsigned c0 = __hip_atomic_load(cnt, __ATOMIC_RELAXED, __HIP_MEMORY_SCOPE_AGENT);
    asm volatile("s_waitcnt lgkmcnt(0)" ::: "memory");
    SB();
    __builtin_amdgcn_s_barrier();
    SB();
    MEMBAR();
    const int iw = wave & 1;
    const int v = (wave < 4) ? (wave >> 1) : 3 - ((wave >> 1) & 1);
    const int lag = (v == 0) ? 0 : (v == 1) ? LAG1 : (v == 2) ? LAG2 : LAG3;
    const unsigned cls = (unsigned)(CLS_FWD >> (16 * v));
    const unsigned call = __builtin_amdgcn_readfirstlane(c0) + 1;
    auto ra = __builtin_amdgcn_make_buffer_rsrc((void*)(myws + OFF_A + (call & 1) * A_BYTES + ((mh * 4 + kq) * 4 + v) * 131072), 0, AREC, 0x00020000);
    const int va = lane * 16 + iw * 2048;
    f32x4 acc[2][8];
#pragma unroll
    for (int i = 0; i < 2; i++)
#pragma unroll
      for (int q = 0; q < 8; q++) acc[i][q] = f32x4{0, 0, 0, 0};
    u32x4 ab[4][2], bf[8];
    int wlim = 0;
    int alim = 4;
    auto wgate = [&](int t) {
      if (t >= wlim) {
        do {
          const int w = min(lds_ld(ctl + C_W0), lds_ld(ctl + C_W1));
          wlim = (w >= 32) ? 64 : w - lag;
          if (t >= wlim) __builtin_amdgcn_s_sleep(1);
        } while (t >= wlim);
      }
      MEMBAR();
    };
    auto agate = [&](int t) {
      if (t >= alim) {
        const int s = (alim - 4) / 7;
        lds_wait_ge(ctl + C_AST, ((cls >> (4 * s)) & 15) + 2);
        alim = (s == 3) ? 64 : alim + 7;
      }
      MEMBAR();
    };
    wgate(0);
    lds_wait_ge(ctl + C_AST, 1);
    if (iw == 0) STAMP(1 + v);
    if (wave == 0) CSTAMP(24);
#pragma unroll
    for (int d = 0; d < ALOOK; d++) {
      ab[d][0] = __builtin_amdgcn_raw_buffer_load_b128(ra, va, d * 4096, 17); SB();
      ab[d][1] = __builtin_amdgcn_raw_buffer_load_b128(ra, va + 1024, d * 4096, 17); SB();
    }
#pragma unroll
    for (int q = 0; q < 8; q++) { bf[q] = *(LAS const u32x4*)(smem + lane * 16 + q * 1024); SB(); }
    int slot = 8192;
#pragma nounroll
    for (int t0 = 0; t0 < 32; t0 += 4) {
#pragma unroll
      for (int u = 0; u < 4; u++) {
        const int t = t0 + u;
        if (ALOOK < 4) {
          agate(t + ALOOK);
          SB();
          ab[(u + ALOOK) & 3][0] = __builtin_amdgcn_raw_buffer_load_b128(ra, va, (t + ALOOK) * 4096, 17); SB();
          ab[(u + ALOOK) & 3][1] = __builtin_amdgcn_raw_buffer_load_b128(ra, va + 1024, (t + ALOOK) * 4096, 17); SB();
        }
        wgate(t + 1);
        LAS const u32x4* wp = (LAS const u32x4*)(smem + lane * 16 + slot);
        slot = (slot == (RING - 1) * 8192) ? 0 : slot + 8192;
        SB();
#pragma unroll
        for (int q = 0; q < 8; q++) {
          MFMA(acc[0][q], ab[u][0], bf[q]);
          MFMA(acc[1][q], ab[u][1], bf[q]);
          SB();
          bf[q] = wp[q * 64];
          SB();
        }
        lds_st(ctl + C_DONE + wave, t + 1);
        SB();
        if (ALOOK == 4) {
          agate(t + 4);
          SB();
          ab[u][0] = __builtin_amdgcn_raw_buffer_load_b128(ra, va, (t + 4) * 4096, 17); SB();
          ab[u][1] = __builtin_amdgcn_raw_buffer_load_b128(ra, va + 1024, (t + 4) * 4096, 17); SB();
        }
      }
    }
    if (iw == 0) STAMP(5 + v);
    if (wave == 0) CSTAMP(25);
    const unsigned mode = lds_wait_ge(ctl + C_MODE, 1);
    const int j = (kq + 1 + v) & 3;
    auto ro = __builtin_amdgcn_make_buffer_rsrc((void*)p.out, 0, MT * N * 4, 0x00020000);
    auto rc = __builtin_amdgcn_make_buffer_rsrc((void*)(myws + OFF_CFLAG), 0, 65536, 0x00020000);
    const int vo = (mh * 256 + j * 64 + iw * 32 + (lane >> 4) * 4) * (N * 4) + nt * 512 + (lane & 15) * 16;
    const int fo = (((nt * 2 + mh) * 4 + j) * 2 + iw) * 128;
    asm volatile("s_nop 7");
    SB();
    if (mode == 1) chain_hop<0, 2>(acc, v, ro, vo, rc, fo, call, tmo, lane);
    else chain_hop<16, 16>(acc, v, ro, vo, rc, fo, call, tmo, lane);
    if (wave == 4 && lane == 0) __hip_atomic_store(cnt, call, __ATOMIC_RELAXED, __HIP_MEMORY_SCOPE_AGENT);
    return;
  }

  const int h = wave - 8;
  if (h < 2) {
    auto rw = __builtin_amdgcn_make_buffer_rsrc((void*)p.w, 0, K * N * 4, 0x00020000);
    const int vw = ((lane >> 4) * 8 + kq * 32) * (N * 4) + (lane & 15) * 16 + nt * 512 + h * 256;
    __builtin_amdgcn_s_setprio(LPRIO);
    __builtin_amdgcn_s_barrier();
    MEMBAR();
    auto rowblk = [&](int t) {
      const int u = t - 4, s = (u * 37) >> 8;
      const int a = rank * 4 + t, b = ((rank + 1 + u + s) & 7) * 4 + s;
      int r; asm("s_cmp_lt_i32 %1, 4\n\ts_cselect_b32 %0, %2, %3" : "=s"(r) : "s"(t), "s"(a), "s"(b) : "scc");
      return r;
    };
    const int rx = mh * ROT;
    u32x4 wb[WD][8];
#pragma unroll
    for (int d = 0; d < WD; d++) {
      const int so = rowblk(d ^ rx) * (128 * N * 4);
#pragma unroll
      for (int i = 0; i < 8; i++) { wb[d][i] = __builtin_amdgcn_raw_buffer_load_b128(rw, vw, so + i * (N * 4), 0); SB(); }
    }
    int freed = RING;
#pragma nounroll
    for (int q = 0; q < 32; q += WD) {
#pragma unroll
      for (int d = 0; d < WD; d++) {
        const int tn = q + d + WD;
        const int so = rowblk((tn & 31) ^ rx) * (128 * N * 4);
        const int ve = (tn < 32) ? vw : 0x40000000;
        u32x4 o[4];
#pragma unroll
        for (int i = 0; i < 4; i++) {
          const f32x4 w0 = __builtin_bit_cast(f32x4, wb[d][2 * i]), w1 = __builtin_bit_cast(f32x4, wb[d][2 * i + 1]);
#pragma unroll
          for (int j = 0; j < 4; j++) o[j][i] = cvtpk_pin(w0[j], w1[j]);
          SB();
          wb[d][2 * i] = __builtin_amdgcn_raw_buffer_load_b128(rw, ve, so + (2 * i) * (N * 4), 0); SB();
          wb[d][2 * i + 1] = __builtin_amdgcn_raw_buffer_load_b128(rw, ve, so + (2 * i + 1) * (N * 4), 0); SB();
        }
        const int t = (q + d) ^ rx;
        if (t >= freed) {
          for (;;) {
            const unsigned dn = __hip_atomic_load(ctl + C_DONE + (lane & 7), __ATOMIC_RELAXED, __HIP_MEMORY_SCOPE_WORKGROUP);
            if (__ballot((int)dn + RING <= t) == 0) break;
            __builtin_amdgcn_s_sleep(1);
          }
          freed = t + 1;
        }
        MEMBAR();
        LAS u32x4* wd = (LAS u32x4*)(smem + (t >= RING ? t - RING : t) * 8192 + h * 4096 + lane * 16);
#pragma unroll
        for (int j = 0; j < 4; j++) wd[j * 64] = o[j];
        asm volatile("s_waitcnt lgkmcnt(0)" ::: "memory");
        if (ROT == 0) lds_st(ctl + h, q + d + 1);
        else { const int n = q + d + 1; lds_st(ctl + h, rx == 0 ? n : ((n & (2 * ROT - 1)) > ROT && false ? n : (n & ~(2 * ROT - 1)))); }
        SB();
      }
    }
    STAMP(13 + h);
  } else if (h == 2) {
    const int dst = (rank + bid) & 7, sl = idx;
    const int c = sl >> 1, smh = sl & 1;
    const int vs = (int)(CLS_INV >> (4 * c)) & 15, gv = vs >> 2, gs = vs & 3;
    const bool local = (dst == rank);
    const unsigned ts = (unsigned)__builtin_amdgcn_s_memrealtime();
    const unsigned tslot = local ? 0 : SLOT0 + (sl * SLOTLEN) / 8;
    const bool early = tslot <= XLEAD;
    const float* xr = p.x + (size_t)(dst * MT + smh * 256 + (lane & 15)) * KS + (lane >> 4) * 8 + gs * 128;
    const unsigned c0 = __hip_atomic_load(cnt, __ATOMIC_RELAXED, __HIP_MEMORY_SCOPE_AGENT);
    SB();
    if (!early) {
      __builtin_amdgcn_s_barrier();
      while ((unsigned)__builtin_amdgcn_s_memrealtime() - ts + XLEAD < tslot) __builtin_amdgcn_s_sleep(2);
    }
    MEMBAR();
    SB();
    f32x4 xv[32];
#pragma unroll
    for (int u = 0; u < 16; u++) {
      const float* q = xr + ((((u >> 2) + 1 + gv) & 3) * 64 + (u & 3) * 16) * KS + (u >> 2) * 32;
      xv[u * 2] = *(const f32x4*)q;
      xv[u * 2 + 1] = *(const f32x4*)(q + 4);
      SB();
    }
    if (early) __builtin_amdgcn_s_barrier();
    SB();
    const unsigned call = __builtin_amdgcn_readfirstlane(c0) + 1;
    char* dbase = p.ws[dst];
    const int tpos = local ? gs : 4 + 7 * gs + ((rank - dst - 1) & 7);
    const unsigned a0 = OFF_A + (call & 1) * A_BYTES + (smh * 16 + gv) * 131072 + tpos * 4096 + lane * 16;
    const unsigned fo = OFF_AFLAG + (smh * 16 + c) * 128 + rank * 4;
    u32x4 o[16];
#pragma unroll
    for (int u = 0; u < 16; u++) {
      o[u] = u32x4{cvtpk(xv[2 * u][0], xv[2 * u][1]), cvtpk(xv[2 * u][2], xv[2 * u][3]), cvtpk(xv[2 * u + 1][0], xv[2 * u + 1][1]), cvtpk(xv[2 * u + 1][2], xv[2 * u + 1][3])};
      asm volatile("" : "+v"(o[u]));
    }
    STAMP(15);
    while ((unsigned)__builtin_amdgcn_s_memrealtime() - ts < tslot) __builtin_amdgcn_s_sleep(1);
#pragma unroll
    for (int u = 0; u < 16; u++)
      asm volatile("global_store_dwordx4 %0, %1, %2 offset:%3 sc0 sc1\n\ts_nop 1" :: "v"(a0 + (u >> 2) * 524288), "v"(o[u]), "s"(dbase), "i"((u & 3) * 1024) : "memory");
    asm volatile("s_waitcnt vmcnt(0)" ::: "memory");
    asm volatile("s_mov_b64 exec, 1\n\tglobal_store_dword %0, %1, %2 sc0 sc1\n\ts_mov_b64 exec, -1" :: "v"(fo), "v"(call), "s"(dbase) : "memory");
    STAMP(16);
    GAS unsigned* xt = (GAS unsigned*)(myws + OFF_XTAB);
    const unsigned long long t0 = __builtin_amdgcn_s_memrealtime();
    unsigned xe;
    for (;;) {
      xe = __hip_atomic_load(xt + ((bid & ~24) | ((lane & 3) << 3)), __ATOMIC_RELAXED, __HIP_MEMORY_SCOPE_AGENT);
      if (__ballot((xe >> 4) != call) == 0 || expired(tmo, call, t0)) break;
      __builtin_amdgcn_s_sleep(4);
    }
    const bool same = __ballot(xe != (unsigned)__builtin_amdgcn_readfirstlane(xe)) == 0;
    lds_st(ctl + C_MODE, same ? 1 : 2);
    if (TOUCH_AT >= 0) {
      lds_wait_ge(ctl + C_W0, TOUCH_AT);
      auto ro = __builtin_amdgcn_make_buffer_rsrc((void*)p.out, 0, MT * N * 4, 0x00020000);
      unsigned sink = 0;
#pragma unroll
      for (int r = 0; r < 4; r++) sink ^= __builtin_amdgcn_raw_buffer_load_b32(ro, (mh * 256 + kq * 64 + r * 16 + (lane >> 2)) * (N * 4) + nt * 512 + (lane & 3) * 128, 0, 16);
      asm volatile("" :: "v"(sink));
    }
  } else {
    const unsigned c0 = __hip_atomic_load(cnt, __ATOMIC_RELAXED, __HIP_MEMORY_SCOPE_AGENT);
    __builtin_amdgcn_s_barrier();
    MEMBAR();
    const unsigned call = __builtin_amdgcn_readfirstlane(c0) + 1;
    const unsigned xcc = __builtin_amdgcn_s_getreg((20 << 0) | (0 << 6) | (3 << 11));
    if (lane == 0) __hip_atomic_store((GAS unsigned*)(myws + OFF_XTAB) + bid, call * 16 + xcc, __ATOMIC_RELAXED, __HIP_MEMORY_SCOPE_AGENT);
    auto rf = __builtin_amdgcn_make_buffer_rsrc((void*)(myws + OFF_AFLAG + mh * 2048), 0, 2048, 0x00020000);
    const unsigned long long t0 = __builtin_amdgcn_s_memrealtime();
    __builtin_amdgcn_s_sleep(POLL_FIRST);
    bool dead = false;
    while (__ballot(__builtin_amdgcn_raw_buffer_load_b32(rf, (lane & 15) * 128 + rank * 4, 0, 17) < call) != 0) {
      MEMBAR();
      if (expired(tmo, call, t0)) { dead = true; break; }
      __builtin_amdgcn_s_sleep(POLL_SLEEP);
    }
    MEMBAR();
    lds_st(ctl + C_AST, 1);
    STAMP(17);
    int done = 0;
    while (done < 16 && !dead) {
      const int cl = min(done + ((lane >> 3) & 3), 15);
      const unsigned long long miss = __ballot(__builtin_amdgcn_raw_buffer_load_b32(rf, cl * 128 + (lane & 7) * 4, 0, 17) < call);
      MEMBAR();
      const int adv = __builtin_ctzll(miss | (1ull << 32)) >> 3;
      if (adv) {
        if (done == 0) STAMP(18);
        done = min(done + adv, 16);
        lds_st(ctl + C_AST, done + 1);
      } else if (expired(tmo, call, t0)) dead = true;
      if (adv < 4) __builtin_amdgcn_s_sleep(POLL_SLEEP);
    }
    lds_st(ctl + C_AST, 17);
    STAMP(19);
  }
}

extern "C" void kernel_launch(int rank, int nranks, ncclComm_t comm, hipStream_t stream,
                              void* const* d_in, const int* in_sizes, int n_in,
                              void* d_out, int out_size, const int* in_dtypes, int out_dtype,
                              void* const* d_out_all, void* const* d_ws_all, size_t ws_bytes) {
  if (nranks != NR || (unsigned)rank >= (unsigned)NR || ws_bytes < WS_NEED) { fprintf(stderr, "bad config\n"); return; }
  static bool init = false;
  if (!init) { HIP_CHECK(hipFuncSetAttribute((const void*)a2a_gemm, hipFuncAttributeMaxDynamicSharedMemorySize, LDS_BYTES)); init = true; }
  Params p;
  p.x = (const float*)d_in[0]; p.w = (const float*)d_in[1]; p.out = (float*)d_out; p.rank = rank;
  for (int i = 0; i < NR; i++) p.ws[i] = (char*)d_ws_all[i];
  hipLaunchKernelGGL(a2a_gemm, dim3(256), dim3(768), LDS_BYTES, stream, p);
  HIP_CHECK(hipGetLastError());
}
```

```cpp
#include <hip/hip_runtime.h>
#include <rccl/rccl.h>
#include <cstdio>

#define HIP_CHECK(expr) do { hipError_t _e = (expr); if (_e != hipSuccess) { \
    fprintf(stderr, "HIP error %s at %s:%d\n", hipGetErrorString(_e), __FILE__, __LINE__); \
    return; } } while (0)

typedef unsigned u32x4 __attribute__((ext_vector_type(4)));
typedef unsigned u32x8 __attribute__((ext_vector_type(8)));
typedef int i32x4 __attribute__((ext_vector_type(4)));
typedef float f32x4 __attribute__((ext_vector_type(4)));
#define LAS __attribute__((address_space(3)))
#define GAS __attribute__((address_space(1)))
#define SB() __builtin_amdgcn_sched_barrier(0)
#define MEMBAR() asm volatile("" ::: "memory")

constexpr int NR = 8, MT = 512, N = 4096, K = 4096, KS = 512;
constexpr unsigned A_BYTES = MT * K * 2;
constexpr unsigned OFF_A = 0;
constexpr unsigned OFF_AFLAG = 2 * A_BYTES;
constexpr unsigned OFF_CFLAG = OFF_AFLAG + 4096;
constexpr unsigned OFF_XTAB = OFF_CFLAG + 131072;
constexpr unsigned OFF_TMO = OFF_XTAB + 4096;
constexpr unsigned OFF_CNT = OFF_TMO + 4096;
constexpr unsigned WS_NEED = OFF_CNT + 4096;
constexpr unsigned DEADLINE = 50000000u;

#define RING 19
#define WFL 12
#define ALOOK 2
#define FAST0 1
#define NSEND 1
#define SLOT0 150
#define SLOTLEN 280
#define XLEAD 400
#define XMAX 5000
#define POLL_SLEEP 3
#define LPRIO 3
#define SPRIO 3
#define AREC 1
#define DMAP 1
#define ROT 1

#define TLBW 0
constexpr int LDS_CTL = RING * 8192, LDS_BYTES = LDS_CTL + 256;
static_assert(LDS_BYTES <= 163840, "LDS");
enum { C_W0 = 0, C_W1 = 1, C_AST = 2, C_MODE = 3, C_CALL = 4, C_DONE = 8 };

struct Params { const float* x; const float* w; float* out; char* myws; char* ws[NR]; int rank; };

#define STAMP(i)
#define PUT(i, val)
#define NOW() 0ull

__device__ __forceinline__ unsigned cvtpk(float a, float b) { unsigned r; asm volatile("v_cvt_pk_f16_f32 %0, %1, %2" : "=v"(r) : "v"(a), "v"(b)); return r; }
__device__ __forceinline__ unsigned lds_ldv(LAS unsigned* p) { return __hip_atomic_load(p, __ATOMIC_RELAXED, __HIP_MEMORY_SCOPE_WORKGROUP); }
__device__ __forceinline__ unsigned lds_ld(LAS unsigned* p) { return __builtin_amdgcn_readfirstlane(lds_ldv(p)); }
__device__ __forceinline__ void lds_st(LAS unsigned* p, unsigned v) { __hip_atomic_store(p, v, __ATOMIC_RELAXED, __HIP_MEMORY_SCOPE_WORKGROUP); }
__device__ __forceinline__ unsigned lds_wait_ge(LAS unsigned* p, unsigned v) {
  unsigned r;
  while ((r = lds_ld(p)) < v) __builtin_amdgcn_s_sleep(1);
  MEMBAR();
  return r;
}
__device__ __forceinline__ int min8(int x) {
  x = min(x, __builtin_amdgcn_update_dpp(x, x, 0xb1, 0xf, 0xf, false));
  x = min(x, __builtin_amdgcn_update_dpp(x, x, 0x4e, 0xf, 0xf, false));
  return min(__builtin_amdgcn_readlane(x, 0), __builtin_amdgcn_readlane(x, 4));
}
__device__ __forceinline__ bool expired(GAS unsigned* tmo, unsigned call, unsigned t0) {
  const unsigned dt = (unsigned)__builtin_amdgcn_s_memrealtime() - t0;
  if (dt < 20000) return false;
  if (dt > DEADLINE || __builtin_amdgcn_readfirstlane(__hip_atomic_load(tmo, __ATOMIC_RELAXED, __HIP_MEMORY_SCOPE_AGENT)) == call) {
    __hip_atomic_store(tmo, call, __ATOMIC_RELAXED, __HIP_MEMORY_SCOPE_AGENT);
    return true;
  }
  return false;
}
__device__ __forceinline__ unsigned sload(const void* p) { unsigned r; asm volatile("s_load_dword %0, %1, 0x0 glc\n\ts_waitcnt lgkmcnt(0)" : "=s"(r) : "s"(p) : "memory"); return r; }
__device__ __forceinline__ u32x4 sload4(const void* p) { u32x4 r; asm volatile("s_load_dwordx4 %0, %1, 0x0 glc\n\ts_waitcnt lgkmcnt(0)" : "=s"(r) : "s"(p) : "memory"); return r; }
__device__ __forceinline__ u32x8 sload8(const void* p) { u32x8 r; asm volatile("s_load_dwordx8 %0, %1, 0x0 glc\n\ts_waitcnt lgkmcnt(0)" : "=s"(r) : "s"(p) : "memory"); return r; }
#define MFMA(acc, a, b) asm volatile("v_mfma_f32_16x16x32_f16 %0, %1, %2, %0" : "+v"(acc) : "v"(a), "v"(b))
#define WAITV(n) asm volatile("s_waitcnt vmcnt(%0)" :: "i"(n) : "memory")

template <int STA, int LDA>
__device__ __forceinline__ void ring_reduce(f32x4 (&acc)[2][8], float* out, int vo, int kq, __amdgpu_buffer_rsrc_t rc, int fo,
                                            unsigned call, GAS unsigned* tmo, int lane, unsigned t0) {
  auto ro = __builtin_amdgcn_make_buffer_rsrc((void*)out, 0, MT * N * 4, 0x00020000);
#pragma unroll
  for (int v = 0; v < 4; v++) {
    const int i = v >> 1, c0 = (v & 1) * 4;
    const int rg = (i + 1 + (kq >> 1)) & 1, hf = (v ^ kq) & 1;
    const int so = rg * 16 * (N * 4) + hf * 256, fv = fo + (rg * 2 + hf) * 64;
    f32x4 o[4];
    if (v > 0) {
      int spins = 0;
      while (__builtin_amdgcn_readfirstlane(__builtin_amdgcn_raw_buffer_load_b32(rc, 0, fv, LDA)) < call * 4 + v) {
        MEMBAR();
        if (++spins > 64) { if (expired(tmo, call, t0)) break; __builtin_amdgcn_s_sleep(2); }
      }
      MEMBAR();
#pragma unroll
      for (int e = 0; e < 4; e++) { o[e] = __builtin_bit_cast(f32x4, __builtin_amdgcn_raw_buffer_load_b128(ro, vo, so + e * (N * 4), LDA)); SB(); }
    }
    if (v == 3 && __hip_atomic_load(tmo, __ATOMIC_RELAXED, __HIP_MEMORY_SCOPE_AGENT) == call) {
      const float q = __builtin_nanf("");
#pragma unroll
      for (int e = 0; e < 4; e++) o[e] = f32x4{q, q, q, q};
    }
#pragma unroll
    for (int e = 0; e < 4; e++) {
#pragma unroll
      for (int c = 0; c < 4; c++) {
        float t_;
        if (v > 0) { t_ = o[e][c]; asm volatile("v_add_f32 %0, %0, %1" : "+v"(t_) : "v"(acc[i][c0 + c][e])); }
        else t_ = acc[i][c0 + c][e];
        o[e][c] = t_;
      }
      asm volatile("" : "+v"(o[e]));
      __builtin_amdgcn_raw_buffer_store_b128(__builtin_bit_cast(u32x4, o[e]), ro, vo, so + e * (N * 4), STA);
      SB();
      asm volatile("s_nop 1");
      SB();
    }
    if (v < 3) {
      asm volatile("s_waitcnt vmcnt(0)" ::: "memory");
      if (lane == 0) __builtin_amdgcn_raw_buffer_store_b32(call * 4 + v + 1, rc, 0, fv, STA);
    }
    SB();
  }
}

__global__ __launch_bounds__(768, 3) void a2a_gemm(Params p) {
  extern __shared__ __attribute__((aligned(16))) char smem_[];
  LAS char* smem = (LAS char*)smem_;
  LAS unsigned* ctl = (LAS unsigned*)(smem + LDS_CTL);
  const unsigned tstart = (unsigned)__builtin_amdgcn_s_memrealtime();
  const int lane = threadIdx.x & 63;
  const int wave = __builtin_amdgcn_readfirstlane(threadIdx.x >> 6);
  const int bid = blockIdx.x;
  const int rank = p.rank;
  char* myws = p.myws;
  const int idx = bid >> 3;
  const int kq = idx & 3, mh = (idx >> 2) & 1, nt = (bid & 7) * 4 + (idx >> 3);
  GAS unsigned* tmo = (GAS unsigned*)(myws + OFF_TMO);
  if (wave == 0) STAMP(0);

  const int dst = (rank + idx) & 7, sl0 = (idx >> 3) * 8 + (bid & 7), sl = mh ? 31 - sl0 : sl0;
  const bool local = (dst == rank);
  const unsigned tslot = local ? 0 : SLOT0 + (sl * SLOTLEN) / 8;
  const bool early = tslot <= XLEAD;

  if (wave >= 10) {
    const int h = wave - 10;
    auto rw = __builtin_amdgcn_make_buffer_rsrc((void*)p.w, 0, K * N * 4, 0x00020000);
    const int vw = ((lane >> 4) * 8 + kq * 32) * (N * 4) + (lane & 15) * 16 + nt * 512 + ((h ^ kq) & 1) * 256;
    const int rx = mh * ROT;
    const int miss = 2 * ((kq + 1) & 3) + 1;
    auto rowoff = [&](int t) {
      const int u = t - 4, j = u - 7 * ((u * 37) >> 8), i = j + (j >= miss ? 1 : 0);
      const int b = rank + 1 + ((kq - (i >> 1)) & 3) + 4 * (i & 1);
      int src; asm("s_cmp_lt_i32 %1, 4\n\ts_cselect_b32 %0, %2, %3" : "=s"(src) : "s"(t), "s"(rank), "s"(b) : "scc");
      return ((src & 7) * 4 + (t & 3)) * (128 * N * 4);
    };
    __builtin_amdgcn_s_setprio(LPRIO);
    if (!FAST0 || early) __builtin_amdgcn_s_barrier();
    MEMBAR();
    u32x4 wb[16];
    int so0 = (rank * 4 + rx) * (128 * N * 4), so1 = (rank * 4 + (1 ^ rx)) * (128 * N * 4);
#pragma unroll
    for (int n = 0; n < WFL; n++) { wb[n] = __builtin_amdgcn_raw_buffer_load_b128(rw, vw, (n < 8 ? so0 : so1) + (n & 7) * (N * 4), 0); SB(); }
    if (FAST0 && !early) __builtin_amdgcn_s_barrier();
    MEMBAR();
    int freed = RING;
    unsigned long long z0 = 0, z1 = 0, z2 = 0;
#pragma nounroll
    for (int q = 0; q < 32; q += 2) {
      const int so2 = rowoff(((q + 2) & 31) ^ rx), so3 = rowoff(((q + 3) & 31) ^ rx);
      const int ve = (q < 30) ? vw : 0x40000000;
#pragma unroll
      for (int d = 0; d < 2; d++) {
        u32x4 o[4];
#pragma unroll
        for (int i = 0; i < 4; i++) {
          const int m = d * 8 + 2 * i, n = m + WFL;
          const f32x4 w0 = __builtin_bit_cast(f32x4, wb[m]), w1 = __builtin_bit_cast(f32x4, wb[m + 1]);
#pragma unroll
          for (int j = 0; j < 4; j++) o[j][i] = cvtpk(w0[j], w1[j]);
          SB();
          const int sb = (n < 8) ? so0 : (n < 16) ? so1 : (n < 24) ? so2 : so3;
          wb[n & 15] = __builtin_amdgcn_raw_buffer_load_b128(rw, n < 16 ? vw : ve, sb + (n & 7) * (N * 4), 0); SB();
          wb[(n + 1) & 15] = __builtin_amdgcn_raw_buffer_load_b128(rw, n < 16 ? vw : ve, sb + ((n + 1) & 7) * (N * 4), 0); SB();
        }
        const int t = (q + d) ^ rx;
        if (t >= freed) {
          while (__ballot((int)lds_ldv(ctl + C_DONE + (lane & 7)) + RING <= t) != 0) __builtin_amdgcn_s_sleep(1);
          freed = t + 1;
        }
        MEMBAR();
        LAS u32x4* wd = (LAS u32x4*)(smem + (t >= RING ? t - RING : t) * 8192 + h * 4096 + lane * 16);
#pragma unroll
        for (int j = 0; j < 4; j++) wd[j * 64] = o[j];
        MEMBAR();
        lds_st(ctl + h, d ? q + 2 : q + 1 - rx);
        SB();
      }
      so0 = so2; so1 = so3;
      if (q == 6) z0 = NOW(); if (q == 14) z1 = NOW(); if (q == 22) z2 = NOW();
    }
    if (h == 0) { STAMP(13); PUT(20, z0); PUT(21, z1); PUT(22, z2); PUT(14, z2); }
    return;
  }
  if (wave == 9) {
    const int r = (rank - dst - 1) & 7;
    const int gs = sl >> 3, gi = (sl >> 1) & 3, smh = sl & 1, skq = (gi + r) & 3;
    const int i = 2 * gi + (r >> 2);
    const int tpos = local ? gs : 4 + 7 * gs + i - (i > 2 * ((skq + 1) & 3) + 1 ? 1 : 0);
    const float* xr = p.x + (size_t)(dst * MT + smh * 256 + (lane & 15)) * KS + (lane >> 4) * 8 + (tpos & 3) * 128 + skq * 32;
    __builtin_amdgcn_s_setprio(SPRIO);
    SB();
    if (!early) {
      __builtin_amdgcn_s_barrier();
      const unsigned tx = min(tslot - XLEAD, (unsigned)XMAX);
      while ((unsigned)__builtin_amdgcn_s_memrealtime() - tstart < tx) __builtin_amdgcn_s_sleep(2);
    }
    MEMBAR();
    SB();
    f32x4 xv[32];
#pragma unroll
    for (int u = 0; u < 16; u++) {
      xv[u * 2] = *(const f32x4*)(xr + u * 16 * KS);
      xv[u * 2 + 1] = *(const f32x4*)(xr + u * 16 * KS + 4);
      SB();
    }
    if (early) __builtin_amdgcn_s_barrier();
    SB();
    const unsigned call = lds_wait_ge(ctl + C_CALL, 1);
    char* dbase = p.ws[dst];
    const unsigned a0 = OFF_A + (call & 1) * A_BYTES + tpos * 131072 + (smh * 4 + skq) * 16384 + lane * 16;
    const unsigned fo = OFF_AFLAG + ((smh * 4 + skq) * 4 + (tpos >> 3)) * 128 + (tpos & 7) * 4;
    u32x4 o[16];
#pragma unroll
    for (int u = 0; u < 16; u++)
      o[u] = u32x4{cvtpk(xv[2 * u][0], xv[2 * u][1]), cvtpk(xv[2 * u][2], xv[2 * u][3]), cvtpk(xv[2 * u + 1][0], xv[2 * u + 1][1]), cvtpk(xv[2 * u + 1][2], xv[2 * u + 1][3])};
    const unsigned long long q0 = NOW();
    while ((unsigned)__builtin_amdgcn_s_memrealtime() - tstart < tslot) __builtin_amdgcn_s_sleep(1);
#pragma unroll
    for (int u = 0; u < 16; u++)
      asm volatile("global_store_dwordx4 %0, %1, %2 offset:%3 sc0 sc1\n\ts_nop 1" :: "v"(a0 + (u >> 2) * 4096), "v"(o[u]), "s"(dbase), "i"((u & 3) * 1024) : "memory");
    asm volatile("s_waitcnt vmcnt(0)" ::: "memory");
    asm volatile("s_mov_b64 exec, 1\n\tglobal_store_dword %0, %1, %2 sc0 sc1\n\ts_mov_b64 exec, -1" :: "v"(fo), "v"(call), "s"(dbase) : "memory");
    STAMP(16); PUT(15, q0);
    return;
  }
  if (wave == 8) {
    __builtin_amdgcn_s_setprio(SPRIO);
    const unsigned call = sload(myws + OFF_CNT + bid * 4) + 1;
    __builtin_amdgcn_s_barrier();
    MEMBAR();
    lds_st(ctl + C_CALL, call);
    if (lane == 0) __hip_atomic_store((GAS unsigned*)(myws + OFF_CNT) + bid, call, __ATOMIC_RELAXED, __HIP_MEMORY_SCOPE_AGENT);
    const unsigned xcc = __builtin_amdgcn_s_getreg((3 << 11) | 20);
    const char* xt = myws + OFF_XTAB + (nt * 2 + mh) * 64;
    if (lane == 0) __hip_atomic_store((GAS unsigned*)xt + kq, call * 16 + xcc, __ATOMIC_RELAXED, __HIP_MEMORY_SCOPE_AGENT);
    const char* af = myws + OFF_AFLAG + (mh * 4 + kq) * 512;
    while ((unsigned)__builtin_amdgcn_s_memrealtime() - tstart < 220) __builtin_amdgcn_s_sleep(2);
    int got = 0;
    unsigned long long p0 = 0, p1 = 0;
    while (got < 32) {
      const u32x8 f = sload8(af + (got >> 3) * 128);
      unsigned m = 0;
#pragma unroll
      for (int q = 0; q < 8; q++) m |= (f[q] >= call ? 1u : 0u) << q;
      const int n = (got & ~7) + __builtin_ctz(~m);
      if (n > got) {
        if (got == 0) p0 = NOW();
        if (got < 11 && n >= 11) p1 = NOW();
        got = n;
        lds_st(ctl + C_AST, n == 32 ? 64 : n);
        if ((n & 7) == 0) continue;
      } else if (expired(tmo, call, tstart)) break;
      __builtin_amdgcn_s_sleep(POLL_SLEEP);
    }
    lds_st(ctl + C_AST, 64);
    STAMP(19); PUT(17, p0); PUT(18, p1);
    u32x4 xe;
    for (;;) {
      xe = sload4(xt);
      if (((xe[0] >> 4) == call && (xe[1] >> 4) == call && (xe[2] >> 4) == call && (xe[3] >> 4) == call) || expired(tmo, call, tstart)) break;
      __builtin_amdgcn_s_sleep(8);
    }
    lds_st(ctl + C_MODE, (xe[0] == xe[1] && xe[1] == xe[2] && xe[2] == xe[3]) ? 1 : 2);
    return;
  }

  if (TLBW && wave == 1) {
    const char* w0 = (const char*)p.w + (size_t)idx * 2097152 + (bid & 7) * 2048;
    const char* o0 = (const char*)p.out + (idx & 3) * 2097152 + (bid & 7) * 2048;
    const char* a0 = myws + (idx % 5) * 2097152 + 8192 + 2097152 - 65536;
    const char* d0 = p.ws[dst] + OFF_TMO;
    unsigned t0_, t1_, t2_, t3_, t4_, t5_, t6_, t7_;
    if (TLBW == 1)
      asm volatile("s_load_dword %0, %3, 0x0\n\ts_load_dword %1, %4, 0x0\n\ts_load_dword %2, %5, 0x0\n\ts_barrier\n\ts_waitcnt lgkmcnt(0)"
                   : "=&s"(t0_), "=&s"(t1_), "=&s"(t2_) : "s"(w0), "s"(o0), "s"(a0) : "memory");
    else
      asm volatile("s_load_dword %0, %8, 0x0\n\ts_load_dword %1, %9, 0x0\n\ts_load_dword %2, %10, 0x0\n\ts_load_dword %3, %11, 0x0\n\t"
                   "s_load_dword %4, %12, 0x0\n\ts_load_dword %5, %13, 0x0\n\ts_load_dword %6, %14, 0x0\n\ts_load_dword %7, %15, 0x0\n\ts_barrier\n\ts_waitcnt lgkmcnt(0)"
                   : "=&s"(t0_), "=&s"(t1_), "=&s"(t2_), "=&s"(t3_), "=&s"(t4_), "=&s"(t5_), "=&s"(t6_), "=&s"(t7_)
                   : "s"(w0), "s"(o0), "s"(a0), "s"(d0), "s"(d0 - 2097152), "s"(d0 - 2 * 2097152), "s"(d0 - 3 * 2097152), "s"(d0 - 4 * 2097152) : "memory");
  } else {
    if (wave == 0 && lane < 32) lds_st(ctl + lane, 0);
    asm volatile("s_waitcnt lgkmcnt(0)" ::: "memory");
    SB();
    __builtin_amdgcn_s_barrier();
  }
  SB();
  MEMBAR();
  const unsigned call = lds_wait_ge(ctl + C_CALL, 1);
  auto ra = __builtin_amdgcn_make_buffer_rsrc((void*)(myws + OFF_A + (call & 1) * A_BYTES + (mh * 4 + kq) * 16384), 0, AREC ? 31 * 131072 + 16384 : 0, 0x00020000);
  const int r0 = (1 + (kq >> 1)) & 1;
  const int vl = lane * 16;
  const int va0 = vl + wave * 2048 + r0 * 1024, va1 = va0 ^ 1024;
  f32x4 acc[2][8];
#pragma unroll
  for (int i = 0; i < 2; i++)
#pragma unroll
    for (int q = 0; q < 8; q++) { acc[i][q] = f32x4{0, 0, 0, 0}; asm volatile("" : "+v"(acc[i][q])); }
  u32x4 ab[4][2], bf[8];
  int wlim = 0;
  int alim = 0;
  auto wgate = [&](int t) {
    if (t >= wlim) {
      do {
        const int w = min(lds_ld(ctl + C_W0), lds_ld(ctl + C_W1));
        wlim = (w >= 32) ? 64 : w;
        if (t >= wlim) __builtin_amdgcn_s_sleep(1);
      } while (t >= wlim);
    }
    MEMBAR();
  };
  auto agate = [&](int t) {
    if (t >= alim) alim = lds_wait_ge(ctl + C_AST, t + 1);
    MEMBAR();
  };
  wgate(0);
#pragma unroll
  for (int q = 0; q < 8; q++) { bf[q] = *(LAS const u32x4*)(smem + vl + q * 1024); SB(); }
  agate(ALOOK - 1);
  const unsigned long long sb = NOW();
  unsigned long long s8 = 0, s16 = 0, s24 = 0;
#pragma unroll
  for (int d = 0; d < ALOOK; d++) {
    ab[d][0] = __builtin_amdgcn_raw_buffer_load_b128(ra, va0, d * 131072, 17); SB();
    ab[d][1] = __builtin_amdgcn_raw_buffer_load_b128(ra, va1, d * 131072, 17); SB();
  }
  int slot = 8192;
#pragma nounroll
  for (int t0 = 0; t0 < 32; t0 += 4) {
#pragma unroll
    for (int u = 0; u < 4; u++) {
      const int t = t0 + u;
      agate(t + ALOOK);
      SB();
      ab[(u + ALOOK) & 3][0] = __builtin_amdgcn_raw_buffer_load_b128(ra, va0, (t + ALOOK) * 131072, 17); SB();
      ab[(u + ALOOK) & 3][1] = __builtin_amdgcn_raw_buffer_load_b128(ra, va1, (t + ALOOK) * 131072, 17); SB();
      wgate(t + 1);
      LAS const u32x4* wp = (LAS const u32x4*)(smem + vl + slot);
      slot = (slot == (RING - 1) * 8192) ? 0 : slot + 8192;
      SB();
#pragma unroll
      for (int q = 0; q < 8; q++) {
        MFMA(acc[0][q], ab[u][0], bf[q]);
        MFMA(acc[1][q], ab[u][1], bf[q]);
        SB();
        bf[q] = wp[q * 64];
        SB();
      }
      if (u == 3 && t0 < 16) { MEMBAR(); lds_st(ctl + C_DONE + wave, t + 2); }
      SB();
    }
    if (t0 == 4) s8 = NOW(); if (t0 == 12) s16 = NOW(); if (t0 == 20) s24 = NOW();
  }
  const unsigned long long se = NOW();
  const unsigned mode = lds_wait_ge(ctl + C_MODE, 1);
  auto rc = __builtin_amdgcn_make_buffer_rsrc((void*)(myws + OFF_CFLAG), 0, 131072, 0x00020000);
  const int vo = (mh * 256 + wave * 32 + (lane >> 4) * 4) * (N * 4) + nt * 512 + (lane & 15) * 16;
  const int fo = ((nt * 2 + mh) * 8 + wave) * 256;
  asm volatile("s_nop 7");
  SB();
  if (mode == 1) ring_reduce<0, 2>(acc, p.out, vo, kq, rc, fo, call, tmo, lane, tstart);
  else ring_reduce<16, 16>(acc, p.out, vo, kq, rc, fo, call, tmo, lane, tstart);
}

extern "C" void kernel_launch(int rank, int nranks, ncclComm_t comm, hipStream_t stream,
                              void* const* d_in, const int* in_sizes, int n_in,
                              void* d_out, int out_size, const int* in_dtypes, int out_dtype,
                              void* const* d_out_all, void* const* d_ws_all, size_t ws_bytes) {
  if (nranks != NR || (unsigned)rank >= (unsigned)NR || ws_bytes < WS_NEED) { fprintf(stderr, "bad config\n"); return; }
  static bool init = false;
  if (!init) { HIP_CHECK(hipFuncSetAttribute((const void*)a2a_gemm, hipFuncAttributeMaxDynamicSharedMemorySize, LDS_BYTES)); init = true; }
  Params p;
  p.x = (const float*)d_in[0]; p.w = (const float*)d_in[1]; p.out = (float*)d_out; p.rank = rank; p.myws = (char*)d_ws_all[rank];
  for (int i = 0; i < NR; i++) p.ws[i] = (char*)d_ws_all[i];
  hipLaunchKernelGGL(a2a_gemm, dim3(256), dim3(768), LDS_BYTES, stream, p);
  HIP_CHECK(hipGetLastError());
}
```

```cpp
#include <hip/hip_runtime.h>
#include <rccl/rccl.h>
#include <cstdio>

#define HIP_CHECK(expr) do { hipError_t _e = (expr); if (_e != hipSuccess) { \
    fprintf(stderr, "HIP error %s at %s:%d\n", hipGetErrorString(_e), __FILE__, __LINE__); \
    return; } } while (0)

typedef unsigned u32x4 __attribute__((ext_vector_type(4)));
typedef unsigned u32x8 __attribute__((ext_vector_type(8)));
typedef int i32x4 __attribute__((ext_vector_type(4)));
typedef float f32x4 __attribute__((ext_vector_type(4)));
#define LAS __attribute__((address_space(3)))
#define GAS __attribute__((address_space(1)))
#define SB() __builtin_amdgcn_sched_barrier(0)
#define MEMBAR() asm volatile("" ::: "memory")

constexpr int NR = 8, MT = 512, N = 4096, K = 4096, KS = 512;
constexpr unsigned A_BYTES = MT * K * 2;
constexpr unsigned OFF_A = 0;
constexpr unsigned OFF_AFLAG = 2 * A_BYTES;
constexpr unsigned OFF_CFLAG = OFF_AFLAG + 4096;
constexpr unsigned OFF_XTAB = OFF_CFLAG + 131072;
constexpr unsigned OFF_TMO = OFF_XTAB + 4096;
constexpr unsigned OFF_CNT = OFF_TMO + 4096;
constexpr unsigned WS_NEED = OFF_CNT + 4096;
constexpr unsigned DEADLINE = 50000000u;

#define RING 19
#define WFL 16
#define ALOOK 2
#define FAST0 1
#define NSEND 1
#define SLOT0 150
#define SLOTLEN 260
#define XLEAD 400
#define XMAX 5000
#define POLL_SLEEP 3
#define LPRIO 0
#define SPRIO 3
#define AREC 1
#define DMAP 6
#define ROT 0

#define TLBW 0
#define WBL 0
#define FINA 0
#define RLDA 16
#define ATCH 0
#define ETOUCH 2
#define BALD 0
#define CMAP 0
constexpr int HSTR = CMAP ? 512 : 256;
#define WPAT 1
#define YS 0
#define LAG 0
#define RPRIO 0
#define BALP 0
#define BALT 16
#define GSLEEP 1
#define POLL0 300
#define HOLD 0
#define DYN 4
constexpr int LDS_CTL = RING * 8192, LDS_BYTES = LDS_CTL + 512;
static_assert(LDS_BYTES <= 163840, "LDS");
enum { C_W0 = 0, C_W1 = 1, C_AST = 2, C_MODE = 3, C_CALL = 4, C_SENT = 5, C_DONE = 8, C_INV = 16, C_ORD = 64 };

struct Params { const float* x; const float* w; float* out; char* myws; char* ws[NR]; int rank; };

#define STAMP(i)
#define PUT(i, val)
#define NOW() 0ull

__device__ __forceinline__ unsigned cvtpk(float a, float b) { unsigned r; asm volatile("v_cvt_pk_f16_f32 %0, %1, %2" : "=v"(r) : "v"(a), "v"(b)); return r; }
__device__ __forceinline__ unsigned lds_ldv(LAS unsigned* p) { return __hip_atomic_load(p, __ATOMIC_RELAXED, __HIP_MEMORY_SCOPE_WORKGROUP); }
__device__ __forceinline__ unsigned lds_ld(LAS unsigned* p) { return __builtin_amdgcn_readfirstlane(lds_ldv(p)); }
__device__ __forceinline__ void lds_st(LAS unsigned* p, unsigned v) { __hip_atomic_store(p, v, __ATOMIC_RELAXED, __HIP_MEMORY_SCOPE_WORKGROUP); }
__device__ __forceinline__ unsigned lds_wait_ge(LAS unsigned* p, unsigned v) {
  unsigned r;
  while ((r = lds_ld(p)) < v) __builtin_amdgcn_s_sleep(1);
  MEMBAR();
  return r;
}
__device__ __forceinline__ int min8(int x) {
  x = min(x, __builtin_amdgcn_update_dpp(x, x, 0xb1, 0xf, 0xf, false));
  x = min(x, __builtin_amdgcn_update_dpp(x, x, 0x4e, 0xf, 0xf, false));
  return min(__builtin_amdgcn_readlane(x, 0), __builtin_amdgcn_readlane(x, 4));
}
__device__ __forceinline__ bool expired(GAS unsigned* tmo, unsigned call, unsigned t0) {
  const unsigned dt = (unsigned)__builtin_amdgcn_s_memrealtime() - t0;
  if (dt < 20000) return false;
  if (dt > DEADLINE || __builtin_amdgcn_readfirstlane(__hip_atomic_load(tmo, __ATOMIC_RELAXED, __HIP_MEMORY_SCOPE_AGENT)) == call) {
    __hip_atomic_store(tmo, call, __ATOMIC_RELAXED, __HIP_MEMORY_SCOPE_AGENT);
    return true;
  }
  return false;
}
__device__ __forceinline__ unsigned sload(const void* p) { unsigned r; asm volatile("s_load_dword %0, %1, 0x0 glc\n\ts_waitcnt lgkmcnt(0)" : "=s"(r) : "s"(p) : "memory"); return r; }
__device__ __forceinline__ u32x4 sload4(const void* p) { u32x4 r; asm volatile("s_load_dwordx4 %0, %1, 0x0 glc\n\ts_waitcnt lgkmcnt(0)" : "=s"(r) : "s"(p) : "memory"); return r; }
__device__ __forceinline__ u32x8 sload8(const void* p) { u32x8 r; asm volatile("s_load_dwordx8 %0, %1, 0x0 glc\n\ts_waitcnt lgkmcnt(0)" : "=s"(r) : "s"(p) : "memory"); return r; }
#define MFMA(acc, a, b) asm volatile("v_mfma_f32_16x16x32_f16 %0, %1, %2, %0" : "+v"(acc) : "v"(a), "v"(b))
#define WAITV(n) asm volatile("s_waitcnt vmcnt(%0)" :: "i"(n) : "memory")

template <int STA, int LDA>
__device__ __forceinline__ void ring_reduce(f32x4 (&acc)[2][8], float* out, int vo, int kq, __amdgpu_buffer_rsrc_t rc, int fo,
                                            unsigned call, GAS unsigned* tmo, int lane, unsigned t0) {
  auto ro = __builtin_amdgcn_make_buffer_rsrc((void*)out, 0, MT * N * 4, 0x00020000);
#pragma unroll
  for (int v = 0; v < 4; v++) {
    const int i = v >> 1, c0 = (v & 1) * 4;
    const int rg = (i + 1 + (kq >> 1)) & 1, hf = (v ^ kq) & 1;
    const int so = rg * 16 * (N * 4) + hf * HSTR, fv = fo + (rg * 2 + hf) * 64;
    f32x4 o[4];
    if (v > 0) {
      int spins = 0;
      while (__builtin_amdgcn_readfirstlane(__builtin_amdgcn_raw_buffer_load_b32(rc, 0, fv, LDA)) < call * 4 + v) {
        MEMBAR();
        if (++spins > 64) { if (expired(tmo, call, t0)) break; __builtin_amdgcn_s_sleep(2); }
      }
      MEMBAR();
#pragma unroll
      for (int e = 0; e < 4; e++) { o[e] = __builtin_bit_cast(f32x4, __builtin_amdgcn_raw_buffer_load_b128(ro, vo, so + e * (N * 4), LDA)); SB(); }
    }
    if (v == 3 && __hip_atomic_load(tmo, __ATOMIC_RELAXED, __HIP_MEMORY_SCOPE_AGENT) == call) {
      const float q = __builtin_nanf("");
#pragma unroll
      for (int e = 0; e < 4; e++) o[e] = f32x4{q, q, q, q};
    }
#pragma unroll
    for (int e = 0; e < 4; e++) {
#pragma unroll
      for (int c = 0; c < 4; c++) {
        float t_;
        if (v > 0) { t_ = o[e][c]; asm volatile("v_add_f32 %0, %0, %1" : "+v"(t_) : "v"(acc[i][c0 + c][e])); }
        else t_ = acc[i][c0 + c][e];
        o[e][c] = t_;
      }
      asm volatile("" : "+v"(o[e]));
      if (v == 3 && STA == 0) __builtin_amdgcn_raw_buffer_store_b128(__builtin_bit_cast(u32x4, o[e]), ro, vo, so + e * (N * 4), FINA);
      else __builtin_amdgcn_raw_buffer_store_b128(__builtin_bit_cast(u32x4, o[e]), ro, vo, so + e * (N * 4), STA);
      SB();
      asm volatile("s_nop 1");
      SB();
    }
    if (v < 3) {
      asm volatile("s_waitcnt vmcnt(0)" ::: "memory");
      if (lane == 0) __builtin_amdgcn_raw_buffer_store_b32(call * 4 + v + 1, rc, 0, fv, STA);
    }
    SB();
  }
}

template <int STA, int LDA>
__device__ __forceinline__ void ring_reduce2(f32x4 (&acc)[2][8], float* out, int vo, int kq, int y, __amdgpu_buffer_rsrc_t rc, int fo,
                                             unsigned call, GAS unsigned* tmo, int lane, unsigned t0) {
  auto ro = __builtin_amdgcn_make_buffer_rsrc((void*)out, 0, MT * N * 4, 0x00020000);
#pragma unroll
  for (int vv = 0; vv < 2; vv++) {
    const int v = 2 * y + vv;
    const int hf = (vv ^ kq) & 1;
    const int so = hf * HSTR, fv = fo + hf * 128;
    auto ri = __builtin_amdgcn_make_buffer_rsrc((void*)out, 0, (vv > 0 || y) ? MT * N * 4 : 0, 0x00020000);
    if (vv > 0 || y) {
      int spins = 0;
      while (__builtin_amdgcn_readfirstlane(__builtin_amdgcn_raw_buffer_load_b32(rc, 0, fv, LDA)) < call * 4 + v) {
        MEMBAR();
        if (++spins > 64) { if (expired(tmo, call, t0)) break; __builtin_amdgcn_s_sleep(2); }
      }
      MEMBAR();
    }
    f32x4 o[8];
#pragma unroll
    for (int r = 0; r < 8; r++) { o[r] = __builtin_bit_cast(f32x4, __builtin_amdgcn_raw_buffer_load_b128(ri, vo, so + ((r >> 2) * 16 + (r & 3)) * (N * 4), LDA)); SB(); }
    if (vv == 1 && y && __hip_atomic_load(tmo, __ATOMIC_RELAXED, __HIP_MEMORY_SCOPE_AGENT) == call) {
      const float q = __builtin_nanf("");
#pragma unroll
      for (int r = 0; r < 8; r++) o[r] = f32x4{q, q, q, q};
    }
#pragma unroll
    for (int r = 0; r < 8; r++) {
#pragma unroll
      for (int c = 0; c < 4; c++) {
        float t_ = o[r][c];
        asm volatile("v_add_f32 %0, %0, %1" : "+v"(t_) : "v"(acc[r >> 2][vv * 4 + c][r & 3]));
        o[r][c] = t_;
      }
      __builtin_amdgcn_raw_buffer_store_b128(__builtin_bit_cast(u32x4, o[r]), ro, vo, so + ((r >> 2) * 16 + (r & 3)) * (N * 4), STA);
      SB();
    }
    asm volatile("s_nop 1");
    if (vv == 0 || !y) {
      asm volatile("s_waitcnt vmcnt(0)" ::: "memory");
      if (lane == 0) __builtin_amdgcn_raw_buffer_store_b32(call * 4 + v + 1, rc, 0, fv, STA);
    }
    SB();
  }
}

__global__ __launch_bounds__(768, 3) void a2a_gemm(Params p) {
  extern __shared__ __attribute__((aligned(16))) char smem_[];
  LAS char* smem = (LAS char*)smem_;
  LAS unsigned* ctl = (LAS unsigned*)(smem + LDS_CTL);
  const unsigned tstart = (unsigned)__builtin_amdgcn_s_memrealtime();
  const int lane = threadIdx.x & 63;
  const int wave = __builtin_amdgcn_readfirstlane(threadIdx.x >> 6);
  const int bid = blockIdx.x;
  const int rank = p.rank;
  char* myws = p.myws;
  const int idx = bid >> 3;
  const int kq = idx & 3, mh = (idx >> 2) & 1, nt = (bid & 7) * 4 + (idx >> 3);
  GAS unsigned* tmo = (GAS unsigned*)(myws + OFF_TMO);
  if (wave == 0) STAMP(0);

  const int ntl = idx >> 3;
  const int un = DMAP == 5 ? (ntl & 1) * 128 + mh * 64 + (ntl >> 1) * 32 + kq * 8 + (bid & 7)
               : DMAP == 6 ? (ntl & 1) * 128 + ((mh ^ (ntl >> 1)) & 1) * 64 + (kq & 1) * 32 + (kq >> 1) * 16 + (ntl >> 1) * 8 + (bid & 7)
                           : ((ntl & 1) ^ 1) * 128 + mh * 64 + (ntl >> 1) * 32 + kq * 8 + (bid & 7);
  const int sl = un < 32 ? un : ((un - 32) * 293) >> 11;
  const int dst = un < 32 ? rank : (rank + 1 + (un - 32) - 7 * sl) & 7;
  const bool local = (dst == rank);
  const unsigned tslot = local ? 0 : SLOT0 + (sl * SLOTLEN) / 8;
  const bool early = tslot <= XLEAD;

  const int colb = CMAP ? (nt >> 1) * 1024 + ((lane >> 3) & 1) * 256 + (nt & 1) * 128 + (lane & 7) * 16 : nt * 512 + (lane & 15) * 16;

  if (wave >= 10) {
    const int h = wave - 10;
    auto rw = __builtin_amdgcn_make_buffer_rsrc((void*)p.w, 0, K * N * 4, 0x00020000);
    const int vw = WPAT ? ((lane >> 5) * 8 + h * 16 + kq * 32) * (N * 4) + nt * 512 + (lane & 31) * 16
                        : ((lane >> 4) * 8 + kq * 32) * (N * 4) + colb + ((h ^ kq) & 1) * HSTR;
    const int wl = WPAT ? (((lane >> 4) ^ kq) & 1) * 4096 + (h * 2 + (lane >> 5)) * 256 + (lane & 15) * 16 : h * 4096 + lane * 16;
    const int rx = mh * ROT;
    const int miss = 2 * ((kq + 1) & 3) + 1;
    auto rowoff = [&](int t) {
      const int u = t - 4, j = u - 7 * ((u * 37) >> 8), i = j + (j >= miss ? 1 : 0);
      const int b = rank + 1 + ((kq - (i >> 1)) & 3) + 4 * (i & 1);
      int src; asm("s_cmp_lt_i32 %1, 4\n\ts_cselect_b32 %0, %2, %3" : "=s"(src) : "s"(t), "s"(rank), "s"(b) : "scc");
      return ((src & 7) * 4 + (t & 3)) * (128 * N * 4);
    };
    __builtin_amdgcn_s_setprio(LPRIO);
    if (!FAST0 || early) __builtin_amdgcn_s_barrier();
    if (HOLD && local) lds_wait_ge(ctl + C_SENT, 1);
    MEMBAR();
    constexpr int NB = WFL > 16 ? 4 : 2;
    u32x4 wb[NB * 8];
    int so[2 * NB];
#pragma unroll
    for (int d = 0; d < NB; d++) so[d] = (rank * 4 + (d ^ rx)) * (128 * N * 4);
#pragma unroll
    for (int n = 0; n < WFL; n++) { wb[n] = __builtin_amdgcn_raw_buffer_load_b128(rw, vw, so[n >> 3] + (n & 7) * (N * 4), 0); SB(); }
    if (FAST0 && !early) __builtin_amdgcn_s_barrier();
    MEMBAR();
    unsigned long long z0 = 0, z1 = 0, z2 = 0;
#pragma nounroll
    for (int q = 0; q < 32; q += NB) {
#pragma unroll
      for (int d = 0; d < NB; d++) so[NB + d] = rowoff(((q + NB + d) & 31) ^ rx);
      const int ve = (q < 32 - NB) ? vw : 0x40000000;
#pragma unroll
      for (int d = 0; d < NB; d++) {
        u32x4 o[4];
#pragma unroll
        for (int i = 0; i < 4; i++) {
          const int m = d * 8 + 2 * i, n = m + WFL;
          const f32x4 w0 = __builtin_bit_cast(f32x4, wb[m]), w1 = __builtin_bit_cast(f32x4, wb[m + 1]);
#pragma unroll
          for (int j = 0; j < 4; j++) o[j][i] = cvtpk(w0[j], w1[j]);
          SB();
          wb[n % (NB * 8)] = __builtin_amdgcn_raw_buffer_load_b128(rw, n < NB * 8 ? vw : ve, so[n >> 3] + (n & 7) * (N * 4), 0); SB();
          wb[(n + 1) % (NB * 8)] = __builtin_amdgcn_raw_buffer_load_b128(rw, n < NB * 8 ? vw : ve, so[n >> 3] + ((n + 1) & 7) * (N * 4), 0); SB();
        }
        const int t = (q + d) ^ rx;
        if (t >= RING) {
          for (;;) {
            const unsigned x = lds_ldv(ctl + ((lane & 8) ? C_INV + t - RING : C_DONE + (lane & 7)));
            const unsigned e = __builtin_amdgcn_readlane(x, 8);
            if (e != 0 && __ballot(x < e) == 0) break;
            __builtin_amdgcn_s_sleep(1);
          }
        }
        MEMBAR();
        LAS u32x4* wd = (LAS u32x4*)(smem + (t >= RING ? t - RING : t) * 8192 + wl);
#pragma unroll
        for (int j = 0; j < 4; j++) wd[j * 64] = o[j];
        MEMBAR();
        lds_st(ctl + h, (d & 1) ? q + d + 1 : q + d + 1 - rx);
        SB();
      }
#pragma unroll
      for (int d = 0; d < NB; d++) so[d] = so[NB + d];
      if (q + NB == 8) z0 = NOW(); if (q + NB == 16) z1 = NOW(); if (q + NB == 24) z2 = NOW();
    }
    if (ETOUCH) {
      auto ro = __builtin_amdgcn_make_buffer_rsrc((void*)p.out, 0, MT * N * 4, 0x00020000);
      unsigned sink = 0;
#pragma unroll
      for (int q = 0; q < 2; q++) sink ^= __builtin_amdgcn_raw_buffer_load_b32(ro, (mh * 256 + kq * 64 + (h * 2 + q) * 16 + (lane >> 2)) * (N * 4) + (CMAP ? (nt >> 1) * 1024 + (nt & 1) * 128 + (lane & 3) * 256 : nt * 512 + (lane & 3) * 128), 0, 16);
      if (ETOUCH > 1 && lane < 2) sink ^= __hip_atomic_load((GAS unsigned*)(myws + OFF_CFLAG + (nt * 2 + mh) * 2048 + (kq * 4 + h * 2 + lane) * 128), __ATOMIC_RELAXED, __HIP_MEMORY_SCOPE_AGENT);
      asm volatile("" :: "v"(sink));
    }
    if (h == 0) { STAMP(13); PUT(20, z0); PUT(21, z1); PUT(22, z2); PUT(14, z2); }
    return;
  }
  if (wave == 9) {
    const int r = (rank - dst - 1) & 7;
    const int gs = sl >> 3, gi = (sl >> 1) & 3, smh = sl & 1, skq = (gi + r) & 3;
    const int i = 2 * gi + (r >> 2);
    const int tpos = local ? gs : 4 + 7 * gs + i - (i > 2 * ((skq + 1) & 3) + 1 ? 1 : 0);
    const float* xr = p.x + (size_t)(dst * MT + smh * 256 + (lane & 15)) * KS + (lane >> 4) * 8 + (tpos & 3) * 128 + skq * 32;
    __builtin_amdgcn_s_setprio(SPRIO);
    SB();
    if (!early) {
      __builtin_amdgcn_s_barrier();
      const unsigned tx = min(tslot - XLEAD, (unsigned)XMAX);
      while ((unsigned)__builtin_amdgcn_s_memrealtime() - tstart < tx) __builtin_amdgcn_s_sleep(2);
    }
    MEMBAR();
    SB();
    f32x4 xv[32];
#pragma unroll
    for (int u = 0; u < 16; u++) {
      xv[u * 2] = *(const f32x4*)(xr + u * 16 * KS);
      xv[u * 2 + 1] = *(const f32x4*)(xr + u * 16 * KS + 4);
      SB();
    }
    if (early) __builtin_amdgcn_s_barrier();
    SB();
    const unsigned call = lds_wait_ge(ctl + C_CALL, 1);
    char* dbase = p.ws[dst];
    const unsigned a0 = OFF_A + (call & 1) * A_BYTES + tpos * 131072 + (smh * 4 + skq) * 16384 + lane * 16;
    const unsigned fo = OFF_AFLAG + ((smh * 4 + skq) * 4 + (tpos >> 3)) * 128 + (tpos & 7) * 4;
    u32x4 o[16];
#pragma unroll
    for (int u = 0; u < 16; u++)
      o[u] = u32x4{cvtpk(xv[2 * u][0], xv[2 * u][1]), cvtpk(xv[2 * u][2], xv[2 * u][3]), cvtpk(xv[2 * u + 1][0], xv[2 * u + 1][1]), cvtpk(xv[2 * u + 1][2], xv[2 * u + 1][3])};
    const unsigned long long q0 = NOW();
    while ((unsigned)__builtin_amdgcn_s_memrealtime() - tstart < tslot) __builtin_amdgcn_s_sleep(1);
#pragma unroll
    for (int u = 0; u < 16; u++)
      asm volatile("global_store_dwordx4 %0, %1, %2 offset:%3 sc0 sc1\n\ts_nop 1" :: "v"(a0 + (u >> 2) * 4096), "v"(o[u]), "s"(dbase), "i"((u & 3) * 1024) : "memory");
    if (HOLD == 1) lds_st(ctl + C_SENT, 1);
    asm volatile("s_waitcnt vmcnt(0)" ::: "memory");
    asm volatile("s_mov_b64 exec, 1\n\tglobal_store_dword %0, %1, %2 sc0 sc1\n\ts_mov_b64 exec, -1" :: "v"(fo), "v"(call), "s"(dbase) : "memory");
    if (HOLD == 2) lds_st(ctl + C_SENT, 1);
    STAMP(16); PUT(15, q0);
    return;
  }
  if (wave == 8) {
    __builtin_amdgcn_s_setprio(SPRIO);
    const unsigned call = sload(myws + OFF_CNT + bid * 4) + 1;
    __builtin_amdgcn_s_barrier();
    MEMBAR();
    lds_st(ctl + C_CALL, call);
    if (lane == 0) __hip_atomic_store((GAS unsigned*)(myws + OFF_CNT) + bid, call, __ATOMIC_RELAXED, __HIP_MEMORY_SCOPE_AGENT);
    const unsigned xcc = __builtin_amdgcn_s_getreg((3 << 11) | 20);
    const char* xt = myws + OFF_XTAB + (nt * 2 + mh) * 64;
    if (lane == 0) __hip_atomic_store((GAS unsigned*)xt + kq, call * 16 + xcc, __ATOMIC_RELAXED, __HIP_MEMORY_SCOPE_AGENT);
    const char* af = myws + OFF_AFLAG + (mh * 4 + kq) * 512;
    while ((unsigned)__builtin_amdgcn_s_memrealtime() - tstart < POLL0) __builtin_amdgcn_s_sleep(2);
    int got = 0;
    unsigned todo = 0xffffffffu, here = 0;
    unsigned long long p0 = 0, p1 = 0;
    bool dead = false;
    auto look = [&](int line) {
      const u32x8 f = sload8(af + line * 128);
      unsigned m = 0;
#pragma unroll
      for (int q = 0; q < 8; q++) m |= (f[q] >= call ? 1u : 0u) << q;
      here |= m << (line * 8);
    };
    while (todo) {
      const int low = __builtin_ctz(todo);
      unsigned take = 0;
      if (!dead) look(low >> 3);
      const unsigned missing = todo & ~here;
      take = missing ? todo & ((1u << __builtin_ctz(missing)) - 1) : todo;
      if (DYN && !take && low >= 4) {
        const unsigned x = lds_ldv(ctl + ((lane & 8) ? (lane & 1) : C_DONE + (lane & 7)));
        if (__ballot((lane & 8) == 0 && (int)x + DYN > got) != 0) {
          const int lim = min(min((int)__builtin_amdgcn_readlane(x, 8), (int)__builtin_amdgcn_readlane(x, 9)), low + RING);
          unsigned cand = todo & here;
          if (!cand && low < 24) { look((low >> 3) + 1); cand = todo & here; }
          if (!cand && low < 16) { look((low >> 3) + 2); cand = todo & here; }
          if (cand && __builtin_ctz(cand) < lim) take = cand & -cand;
        }
      }
      if (take) {
        const bool mine = lane < 32 && ((take >> lane) & 1);
        const int e = got + __builtin_popcount(take & ((1u << (lane & 31)) - 1));
        if (mine) { lds_st(ctl + C_ORD + e, lane); lds_st(ctl + C_INV + lane, e + 1); }
        MEMBAR();
        if (got == 0) p0 = NOW();
        const int n = got + __builtin_popcount(take);
        if (got < 11 && n >= 11) p1 = NOW();
        got = n; todo &= ~take;
        lds_st(ctl + C_AST, n == 32 ? 64 : n);
        if (todo && (__builtin_ctz(todo) >> 3) != (low >> 3)) continue;
      } else if (expired(tmo, call, tstart)) { dead = true; here = 0xffffffffu; continue; }
      __builtin_amdgcn_s_sleep(POLL_SLEEP);
    }
    STAMP(19); PUT(17, p0); PUT(18, p1);
    u32x4 xe;
    for (;;) {
      xe = sload4(xt);
      if (((xe[0] >> 4) == call && (xe[1] >> 4) == call && (xe[2] >> 4) == call && (xe[3] >> 4) == call) || expired(tmo, call, tstart)) break;
      __builtin_amdgcn_s_sleep(8);
    }
    lds_st(ctl + C_MODE, (xe[0] == xe[1] && xe[1] == xe[2] && xe[2] == xe[3]) ? 1 : 2);
    return;
  }

  if (TLBW && wave == 1) {
    const char* w0 = (const char*)p.w + (size_t)idx * 2097152 + (bid & 7) * 2048;
    const char* o0 = (const char*)p.out + (idx & 3) * 2097152 + (bid & 7) * 2048;
    const char* a0 = myws + (idx % 5) * 2097152 + 8192 + 2097152 - 65536;
    const char* d0 = p.ws[dst] + OFF_TMO;
    unsigned t0_, t1_, t2_, t3_, t4_, t5_, t6_, t7_;
    if (TLBW == 1)
      asm volatile("s_load_dword %0, %3, 0x0\n\ts_load_dword %1, %4, 0x0\n\ts_load_dword %2, %5, 0x0\n\ts_barrier\n\ts_waitcnt lgkmcnt(0)"
                   : "=&s"(t0_), "=&s"(t1_), "=&s"(t2_) : "s"(w0), "s"(o0), "s"(a0) : "memory");
    else
      asm volatile("s_load_dword %0, %8, 0x0\n\ts_load_dword %1, %9, 0x0\n\ts_load_dword %2, %10, 0x0\n\ts_load_dword %3, %11, 0x0\n\t"
                   "s_load_dword %4, %12, 0x0\n\ts_load_dword %5, %13, 0x0\n\ts_load_dword %6, %14, 0x0\n\ts_load_dword %7, %15, 0x0\n\ts_barrier\n\ts_waitcnt lgkmcnt(0)"
                   : "=&s"(t0_), "=&s"(t1_), "=&s"(t2_), "=&s"(t3_), "=&s"(t4_), "=&s"(t5_), "=&s"(t6_), "=&s"(t7_)
                   : "s"(w0), "s"(o0), "s"(a0), "s"(d0), "s"(d0 - 2097152), "s"(d0 - 2 * 2097152), "s"(d0 - 3 * 2097152), "s"(d0 - 4 * 2097152) : "memory");
  } else {
    if (wave == 0) { lds_st(ctl + lane, 0); lds_st(ctl + C_ORD + lane, 32); }
    asm volatile("s_waitcnt lgkmcnt(0)" ::: "memory");
    SB();
    __builtin_amdgcn_s_barrier();
  }
  SB();
  MEMBAR();
  const unsigned call = lds_wait_ge(ctl + C_CALL, 1);
  auto ra = __builtin_amdgcn_make_buffer_rsrc((void*)(myws + OFF_A + (call & 1) * A_BYTES + (mh * 4 + kq) * 16384), 0, AREC ? 31 * 131072 + 16384 : 0, 0x00020000);
  const int r0 = (1 + (kq >> 1)) & 1;
  const int vl = lane * 16;
  const int y = wave >> 2, rw2 = YS ? (wave & 3) * 2 + ((y + r0) & 1) : wave;
  const int va0 = vl + rw2 * 2048 + (YS ? 0 : r0 * 1024), va1 = va0 ^ 1024;
  const int lag = y * LAG;
  f32x4 acc[2][8];
#pragma unroll
  for (int i = 0; i < 2; i++)
#pragma unroll
    for (int q = 0; q < 8; q++) { acc[i][q] = f32x4{0, 0, 0, 0}; asm volatile("" : "+v"(acc[i][q])); }
  u32x4 ab[4][2], bf[8];
  int wlim = 0;
  int alim = 0;
  auto wgate = [&](int t) {
    if (t >= wlim) {
      do {
        const int w = min(lds_ld(ctl + C_W0), lds_ld(ctl + C_W1));
        wlim = (w >= 32) ? 64 : w - lag;
        if (GSLEEP && t >= wlim) __builtin_amdgcn_s_sleep(GSLEEP);
      } while (t >= wlim);
    }
    MEMBAR();
  };
  int ord = 0;
  auto agate = [&](int e) {
    if (e >= alim) { alim = lds_wait_ge(ctl + C_AST, e + 1); ord = lds_ldv(ctl + C_ORD + lane); asm volatile("" : "+v"(ord)); }
    MEMBAR();
    return __builtin_amdgcn_readlane(ord, e);
  };
  wgate(0);
#pragma unroll
  for (int q = 0; q < 8; q++) { bf[q] = *(LAS const u32x4*)(smem + vl + q * 1024); SB(); }
  agate(ALOOK - 1);
  const unsigned long long sb = NOW();
  unsigned long long s8 = 0, s16 = 0, s24 = 0;
#pragma unroll
  for (int d = 0; d < ALOOK; d++) {
    ab[d][0] = __builtin_amdgcn_raw_buffer_load_b128(ra, va0, d * 131072, 17); SB();
    ab[d][1] = __builtin_amdgcn_raw_buffer_load_b128(ra, va1, d * 131072, 17); SB();
  }
#pragma nounroll
  for (int t0 = 0; t0 < 32; t0 += 4) {
    if (BALP == 1 && y && t0 == BALT) __builtin_amdgcn_s_setprio(1);
    if (BALP == 2 && y) { if (t0 & 4) __builtin_amdgcn_s_setprio(1); else __builtin_amdgcn_s_setprio(0); }
#pragma unroll
    for (int u = 0; u < 4; u++) {
      const int t = t0 + u;
      if (BALP == 4 && y) { if (u & 1) __builtin_amdgcn_s_setprio(1); else __builtin_amdgcn_s_setprio(0); }
      if (BALD && (u & 1) == 0 && t0 >= 8) {
        while (__ballot((int)lds_ldv(ctl + C_DONE + (lane & 7)) + BALD < t + 1) != 0) __builtin_amdgcn_s_sleep(1);
        MEMBAR();
      }
      const int pa = agate(t + ALOOK);
      SB();
      ab[(u + ALOOK) & 3][0] = __builtin_amdgcn_raw_buffer_load_b128(ra, va0, pa * 131072, 17); SB();
      ab[(u + ALOOK) & 3][1] = __builtin_amdgcn_raw_buffer_load_b128(ra, va1, pa * 131072, 17); SB();
      const int pw = __builtin_amdgcn_readlane(ord, t + 1);
      wgate(pw);
      LAS const u32x4* wp = (LAS const u32x4*)(smem + vl + (pw >= RING ? pw - RING : pw) * 8192);
      SB();
#pragma unroll
      for (int q = 0; q < 8; q++) {
        MFMA(acc[0][q], ab[u][0], bf[q]);
        MFMA(acc[1][q], ab[u][1], bf[q]);
        SB();
        bf[q] = wp[q * 64];
        SB();
      }
      MEMBAR();
      lds_st(ctl + C_DONE + wave, t + 2);
      SB();
    }
    if (t0 == 4) s8 = NOW(); if (t0 == 12) s16 = NOW(); if (t0 == 20) s24 = NOW();
  }
  const unsigned long long se = NOW();
  const unsigned mode = lds_wait_ge(ctl + C_MODE, 1);
  auto rc = __builtin_amdgcn_make_buffer_rsrc((void*)(myws + OFF_CFLAG), 0, 131072, 0x00020000);
  const int vo = (mh * 256 + rw2 * 32 + (lane >> 4) * 4) * (N * 4) + colb;
  const int fo = ((nt * 2 + mh) * 8 + rw2) * 256;
  if (RPRIO) __builtin_amdgcn_s_setprio(RPRIO);
  asm volatile("s_nop 7");
  SB();
  if (YS) {
    if (mode == 1) ring_reduce2<0, RLDA>(acc, p.out, vo, kq, y, rc, fo, call, tmo, lane, tstart);
    else ring_reduce2<16, 16>(acc, p.out, vo, kq, y, rc, fo, call, tmo, lane, tstart);
  } else {
    if (mode == 1) ring_reduce<0, RLDA>(acc, p.out, vo, kq, rc, fo, call, tmo, lane, tstart);
    else ring_reduce<16, 16>(acc, p.out, vo, kq, rc, fo, call, tmo, lane, tstart);
  }
  if (WBL == 1 && wave == 0) asm volatile("s_waitcnt vmcnt(0)\n\tbuffer_wbl2 sc1" ::: "memory");
  if (WBL == 2 && wave == 0 && (idx & 7) == 0) asm volatile("s_waitcnt vmcnt(0)\n\tbuffer_wbl2 sc1" ::: "memory");
}

extern "C" void kernel_launch(int rank, int nranks, ncclComm_t comm, hipStream_t stream,
                              void* const* d_in, const int* in_sizes, int n_in,
                              void* d_out, int out_size, const int* in_dtypes, int out_dtype,
                              void* const* d_out_all, void* const* d_ws_all, size_t ws_bytes) {
  if (nranks != NR || (unsigned)rank >= (unsigned)NR || ws_bytes < WS_NEED) { fprintf(stderr, "bad config\n"); return; }
  static bool init = false;
  if (!init) { HIP_CHECK(hipFuncSetAttribute((const void*)a2a_gemm, hipFuncAttributeMaxDynamicSharedMemorySize, LDS_BYTES)); init = true; }
  Params p;
  p.x = (const float*)d_in[0]; p.w = (const float*)d_in[1]; p.out = (float*)d_out; p.rank = rank; p.myws = (char*)d_ws_all[rank];
  for (int i = 0; i < NR; i++) p.ws[i] = (char*)d_ws_all[i];
  hipLaunchKernelGGL(a2a_gemm, dim3(256), dim3(768), LDS_BYTES, stream, p);
  HIP_CHECK(hipGetLastError());
}
```

```cpp
#include <hip/hip_runtime.h>
#include <rccl/rccl.h>
#include <cstdio>

#define HIP_CHECK(expr) do { hipError_t _e = (expr); if (_e != hipSuccess) { \
    fprintf(stderr, "HIP error %s at %s:%d\n", hipGetErrorString(_e), __FILE__, __LINE__); \
    return; } } while (0)

typedef unsigned u32x4 __attribute__((ext_vector_type(4)));
typedef unsigned u32x8 __attribute__((ext_vector_type(8)));
typedef int i32x4 __attribute__((ext_vector_type(4)));
typedef float f32x4 __attribute__((ext_vector_type(4)));
#define LAS __attribute__((address_space(3)))
#define GAS __attribute__((address_space(1)))
#define SB() __builtin_amdgcn_sched_barrier(0)
#define MEMBAR() asm volatile("" ::: "memory")

constexpr int NR = 8, MT = 512, N = 4096, K = 4096, KS = 512;
constexpr unsigned A_BYTES = MT * K * 2;
constexpr unsigned OFF_A = 0;
constexpr unsigned OFF_AFLAG = 2 * A_BYTES;
constexpr unsigned OFF_CFLAG = OFF_AFLAG + 4096;
constexpr unsigned OFF_XTAB = OFF_CFLAG + 131072;
constexpr unsigned OFF_TMO = OFF_XTAB + 4096;
constexpr unsigned OFF_CNT = OFF_TMO + 4096;
constexpr unsigned WS_NEED = OFF_CNT + 4096;
constexpr unsigned DEADLINE = 50000000u;

#define RING 19
#define WFL 16
#define ALOOK 2
#define FAST0 1
#define NSEND 1
#define SLOT0 150
#define SLOTLEN 260
#define XLEAD 400
#define XMAX 5000
#define POLL_SLEEP 3
#define LPRIO 0
#define SPRIO 3
#define AREC 1
#define DMAP 6
#define ROT 0

#define TLBW 0
#define WBL 0
#define FINA 0
#define RLDA 16
#define ATCH 0
#define ETOUCH 2
#define BALD 0
#define CMAP 0
constexpr int HSTR = CMAP ? 512 : 256;
#define WPAT 1
#define YS 0
#define LAG 0
#define RPRIO 0
#define BALP 0
#define BALT 16
#define GSLEEP 1
#define POLL0 300
#define HOLD 0
#define DYN 4
#define CPF 12288
#define EARLY 88
constexpr int LDS_CTL = RING * 8192, LDS_BYTES = LDS_CTL + 512;
static_assert(LDS_BYTES <= 163840, "LDS");
enum { C_W0 = 0, C_W1 = 1, C_AST = 2, C_MODE = 3, C_CALL = 4, C_SENT = 5, C_DONE = 8, C_INV = 16, C_ORD = 64 };

struct Params { const float* x; const float* w; float* out; char* myws; int rank; int pad; char* ws[NR]; };

#define STAMP(i)
#define PUT(i, val)
#define NOW() 0ull

__device__ __forceinline__ unsigned cvtpk(float a, float b) { unsigned r; asm volatile("v_cvt_pk_f16_f32 %0, %1, %2" : "=v"(r) : "v"(a), "v"(b)); return r; }
__device__ __forceinline__ unsigned lds_ldv(LAS unsigned* p) { return __hip_atomic_load(p, __ATOMIC_RELAXED, __HIP_MEMORY_SCOPE_WORKGROUP); }
__device__ __forceinline__ unsigned lds_ld(LAS unsigned* p) { return __builtin_amdgcn_readfirstlane(lds_ldv(p)); }
__device__ __forceinline__ void lds_st(LAS unsigned* p, unsigned v) { __hip_atomic_store(p, v, __ATOMIC_RELAXED, __HIP_MEMORY_SCOPE_WORKGROUP); }
__device__ __forceinline__ unsigned lds_wait_ge(LAS unsigned* p, unsigned v) {
  unsigned r;
  while ((r = lds_ld(p)) < v) __builtin_amdgcn_s_sleep(1);
  MEMBAR();
  return r;
}
__device__ __forceinline__ int min8(int x) {
  x = min(x, __builtin_amdgcn_update_dpp(x, x, 0xb1, 0xf, 0xf, false));
  x = min(x, __builtin_amdgcn_update_dpp(x, x, 0x4e, 0xf, 0xf, false));
  return min(__builtin_amdgcn_readlane(x, 0), __builtin_amdgcn_readlane(x, 4));
}
__device__ __forceinline__ bool expired(GAS unsigned* tmo, unsigned call, unsigned t0) {
  const unsigned dt = (unsigned)__builtin_amdgcn_s_memrealtime() - t0;
  if (dt < 20000) return false;
  if (dt > DEADLINE || __builtin_amdgcn_readfirstlane(__hip_atomic_load(tmo, __ATOMIC_RELAXED, __HIP_MEMORY_SCOPE_AGENT)) == call) {
    __hip_atomic_store(tmo, call, __ATOMIC_RELAXED, __HIP_MEMORY_SCOPE_AGENT);
    return true;
  }
  return false;
}
__device__ __forceinline__ unsigned sload(const void* p) { unsigned r; asm volatile("s_load_dword %0, %1, 0x0 glc\n\ts_waitcnt lgkmcnt(0)" : "=s"(r) : "s"(p) : "memory"); return r; }
__device__ __forceinline__ u32x4 sload4(const void* p) { u32x4 r; asm volatile("s_load_dwordx4 %0, %1, 0x0 glc\n\ts_waitcnt lgkmcnt(0)" : "=s"(r) : "s"(p) : "memory"); return r; }
__device__ __forceinline__ u32x8 sload8(const void* p) { u32x8 r; asm volatile("s_load_dwordx8 %0, %1, 0x0 glc\n\ts_waitcnt lgkmcnt(0)" : "=s"(r) : "s"(p) : "memory"); return r; }
#define MFMA(acc, a, b) asm volatile("v_mfma_f32_16x16x32_f16 %0, %1, %2, %0" : "+v"(acc) : "v"(a), "v"(b))
#define WAITV(n) asm volatile("s_waitcnt vmcnt(%0)" :: "i"(n) : "memory")

template <int STA, int LDA>
__device__ __forceinline__ void ring_reduce(f32x4 (&acc)[2][8], float* out, int vo, int kq, __amdgpu_buffer_rsrc_t rc, int fo,
                                            unsigned call, GAS unsigned* tmo, int lane, unsigned t0) {
  auto ro = __builtin_amdgcn_make_buffer_rsrc((void*)out, 0, MT * N * 4, 0x00020000);
#pragma unroll
  for (int v = 0; v < 4; v++) {
    const int i = v >> 1, c0 = (v & 1) * 4;
    const int rg = (i + 1 + (kq >> 1)) & 1, hf = (v ^ kq) & 1;
    const int so = rg * 16 * (N * 4) + hf * HSTR, fv = fo + (rg * 2 + hf) * 64;
    f32x4 o[4];
    if (v > 0) {
      int spins = 0;
      while (__builtin_amdgcn_readfirstlane(__builtin_amdgcn_raw_buffer_load_b32(rc, 0, fv, LDA)) < call * 4 + v) {
        MEMBAR();
        if (++spins > 64) { if (expired(tmo, call, t0)) break; __builtin_amdgcn_s_sleep(2); }
      }
      MEMBAR();
#pragma unroll
      for (int e = 0; e < 4; e++) { o[e] = __builtin_bit_cast(f32x4, __builtin_amdgcn_raw_buffer_load_b128(ro, vo, so + e * (N * 4), LDA)); SB(); }
    }
    if (v == 3 && __hip_atomic_load(tmo, __ATOMIC_RELAXED, __HIP_MEMORY_SCOPE_AGENT) == call) {
      const float q = __builtin_nanf("");
#pragma unroll
      for (int e = 0; e < 4; e++) o[e] = f32x4{q, q, q, q};
    }
#pragma unroll
    for (int e = 0; e < 4; e++) {
#pragma unroll
      for (int c = 0; c < 4; c++) {
        float t_;
        if (v > 0) { t_ = o[e][c]; asm volatile("v_add_f32 %0, %0, %1" : "+v"(t_) : "v"(acc[i][c0 + c][e])); }
        else t_ = acc[i][c0 + c][e];
        o[e][c] = t_;
      }
      asm volatile("" : "+v"(o[e]));
      if (v == 3 && STA == 0) __builtin_amdgcn_raw_buffer_store_b128(__builtin_bit_cast(u32x4, o[e]), ro, vo, so + e * (N * 4), FINA);
      else __builtin_amdgcn_raw_buffer_store_b128(__builtin_bit_cast(u32x4, o[e]), ro, vo, so + e * (N * 4), STA);
      SB();
      asm volatile("s_nop 1");
      SB();
    }
    if (v < 3) {
      asm volatile("s_waitcnt vmcnt(0)" ::: "memory");
      if (lane == 0) __builtin_amdgcn_raw_buffer_store_b32(call * 4 + v + 1, rc, 0, fv, STA);
    }
    SB();
  }
}

template <int STA, int LDA>
__device__ __forceinline__ void ring_reduce2(f32x4 (&acc)[2][8], float* out, int vo, int kq, int y, __amdgpu_buffer_rsrc_t rc, int fo,
                                             unsigned call, GAS unsigned* tmo, int lane, unsigned t0) {
  auto ro = __builtin_amdgcn_make_buffer_rsrc((void*)out, 0, MT * N * 4, 0x00020000);
#pragma unroll
  for (int vv = 0; vv < 2; vv++) {
    const int v = 2 * y + vv;
    const int hf = (vv ^ kq) & 1;
    const int so = hf * HSTR, fv = fo + hf * 128;
    auto ri = __builtin_amdgcn_make_buffer_rsrc((void*)out, 0, (vv > 0 || y) ? MT * N * 4 : 0, 0x00020000);
    if (vv > 0 || y) {
      int spins = 0;
      while (__builtin_amdgcn_readfirstlane(__builtin_amdgcn_raw_buffer_load_b32(rc, 0, fv, LDA)) < call * 4 + v) {
        MEMBAR();
        if (++spins > 64) { if (expired(tmo, call, t0)) break; __builtin_amdgcn_s_sleep(2); }
      }
      MEMBAR();
    }
    f32x4 o[8];
#pragma unroll
    for (int r = 0; r < 8; r++) { o[r] = __builtin_bit_cast(f32x4, __builtin_amdgcn_raw_buffer_load_b128(ri, vo, so + ((r >> 2) * 16 + (r & 3)) * (N * 4), LDA)); SB(); }
    if (vv == 1 && y && __hip_atomic_load(tmo, __ATOMIC_RELAXED, __HIP_MEMORY_SCOPE_AGENT) == call) {
      const float q = __builtin_nanf("");
#pragma unroll
      for (int r = 0; r < 8; r++) o[r] = f32x4{q, q, q, q};
    }
#pragma unroll
    for (int r = 0; r < 8; r++) {
#pragma unroll
      for (int c = 0; c < 4; c++) {
        float t_ = o[r][c];
        asm volatile("v_add_f32 %0, %0, %1" : "+v"(t_) : "v"(acc[r >> 2][vv * 4 + c][r & 3]));
        o[r][c] = t_;
      }
      __builtin_amdgcn_raw_buffer_store_b128(__builtin_bit_cast(u32x4, o[r]), ro, vo, so + ((r >> 2) * 16 + (r & 3)) * (N * 4), STA);
      SB();
    }
    asm volatile("s_nop 1");
    if (vv == 0 || !y) {
      asm volatile("s_waitcnt vmcnt(0)" ::: "memory");
      if (lane == 0) __builtin_amdgcn_raw_buffer_store_b32(call * 4 + v + 1, rc, 0, fv, STA);
    }
    SB();
  }
}

__global__ __launch_bounds__(768, 3) void a2a_gemm(Params p) {
  extern __shared__ __attribute__((aligned(16))) char smem_[];
  LAS char* smem = (LAS char*)smem_;
  LAS unsigned* ctl = (LAS unsigned*)(smem + LDS_CTL);
  u32x8 ka; unsigned karank; unsigned long long tclk;
  asm volatile("s_load_dwordx8 %0, %3, 0x0\n\ts_load_dword %1, %3, 0x20\n\ts_memrealtime %2"
               : "=&s"(ka), "=&s"(karank), "=&s"(tclk) : "s"(__builtin_amdgcn_kernarg_segment_ptr()));
  const int lane = threadIdx.x & 63;
  const int wave = __builtin_amdgcn_readfirstlane(threadIdx.x >> 6);
  if (wave == 0) {
    lds_st(ctl + lane, 0); lds_st(ctl + C_ORD + lane, 32);
    asm volatile("s_waitcnt lgkmcnt(0)\n\ts_barrier" ::: "memory");
  } else if (wave < 8) {
    unsigned long long pc; unsigned junk;
    const unsigned off = CPF ? min(((blockIdx.x >> 3) * 7 + wave - 1) * 64u, CPF - 64u) : 0;
    asm volatile("s_getpc_b64 %0\n\ts_load_dword %1, %0, %2\n\ts_barrier\n\ts_waitcnt lgkmcnt(0)" : "=&s"(pc), "=&s"(junk) : "s"(off) : "memory");
  }
  const int bid = blockIdx.x;
  const int idx = bid >> 3;
  const int kq = idx & 3, mh = (idx >> 2) & 1, nt = (bid & 7) * 4 + (idx >> 3);

  const int ntl = idx >> 3;
  const int un = (ntl & 1) * 128 + ((mh ^ (ntl >> 1)) & 1) * 64 + (kq & 1) * 32 + (kq >> 1) * 16 + (ntl >> 1) * 8 + (bid & 7);
  const bool early = un < EARLY;
  int vw0 = ((lane >> 5) * 8 + (wave - 10) * 16 + kq * 32) * (N * 4) + nt * 512 + (lane & 31) * 16;
  asm volatile("s_waitcnt lgkmcnt(0)" : "+s"(ka), "+s"(karank), "+s"(tclk), "+v"(vw0));
  const float* px = (const float*)(((unsigned long long)ka[1] << 32) | ka[0]);
  const float* pw = (const float*)(((unsigned long long)ka[3] << 32) | ka[2]);
  float* pout = (float*)(((unsigned long long)ka[5] << 32) | ka[4]);
  char* myws = (char*)(((unsigned long long)ka[7] << 32) | ka[6]);
  const int rank = karank;
  const unsigned tstart = (unsigned)tclk;
  GAS unsigned* tmo = (GAS unsigned*)(myws + OFF_TMO);
  if (wave == 8) PUT(0, tclk);
  const int colb = CMAP ? (nt >> 1) * 1024 + ((lane >> 3) & 1) * 256 + (nt & 1) * 128 + (lane & 7) * 16 : nt * 512 + (lane & 15) * 16;

  if (__builtin_expect(wave >= 10, 1)) {
    const int h = wave - 10;
    auto rw = __builtin_amdgcn_make_buffer_rsrc((void*)pw, 0, K * N * 4, 0x00020000);
    const int vw = vw0;
    const int wl = WPAT ? (((lane >> 4) ^ kq) & 1) * 4096 + (h * 2 + (lane >> 5)) * 256 + (lane & 15) * 16 : h * 4096 + lane * 16;
    const int rx = mh * ROT;
    const int miss = 2 * ((kq + 1) & 3) + 1;
    auto rowoff = [&](int t) {
      const int u = t - 4, j = u - 7 * ((u * 37) >> 8), i = j + (j >= miss ? 1 : 0);
      const int b = rank + 1 + ((kq - (i >> 1)) & 3) + 4 * (i & 1);
      int src; asm("s_cmp_lt_i32 %1, 4\n\ts_cselect_b32 %0, %2, %3" : "=s"(src) : "s"(t), "s"(rank), "s"(b) : "scc");
      return ((src & 7) * 4 + (t & 3)) * (128 * N * 4);
    };
    __builtin_amdgcn_s_setprio(LPRIO);
    if (!FAST0 || early) __builtin_amdgcn_s_barrier();
    MEMBAR();
    constexpr int NB = WFL > 16 ? 4 : 2;
    u32x4 wb[NB * 8];
    int so[2 * NB];
#pragma unroll
    for (int d = 0; d < NB; d++) so[d] = (rank * 4 + (d ^ rx)) * (128 * N * 4);
#pragma unroll
    for (int n = 0; n < WFL; n++) { wb[n] = __builtin_amdgcn_raw_buffer_load_b128(rw, vw, so[n >> 3] + (n & 7) * (N * 4), 0); SB(); }
    if (FAST0 && !early) __builtin_amdgcn_s_barrier();
    MEMBAR();
    unsigned long long z0 = 0, z1 = 0, z2 = 0;
#pragma nounroll
    for (int q = 0; q < 32; q += NB) {
#pragma unroll
      for (int d = 0; d < NB; d++) so[NB + d] = rowoff(((q + NB + d) & 31) ^ rx);
      const int ve = (q < 32 - NB) ? vw : 0x40000000;
#pragma unroll
      for (int d = 0; d < NB; d++) {
        u32x4 o[4];
#pragma unroll
        for (int i = 0; i < 4; i++) {
          const int m = d * 8 + 2 * i, n = m + WFL;
          const f32x4 w0 = __builtin_bit_cast(f32x4, wb[m]), w1 = __builtin_bit_cast(f32x4, wb[m + 1]);
#pragma unroll
          for (int j = 0; j < 4; j++) o[j][i] = cvtpk(w0[j], w1[j]);
          SB();
          wb[n % (NB * 8)] = __builtin_amdgcn_raw_buffer_load_b128(rw, n < NB * 8 ? vw : ve, so[n >> 3] + (n & 7) * (N * 4), 0); SB();
          wb[(n + 1) % (NB * 8)] = __builtin_amdgcn_raw_buffer_load_b128(rw, n < NB * 8 ? vw : ve, so[n >> 3] + ((n + 1) & 7) * (N * 4), 0); SB();
        }
        const int t = (q + d) ^ rx;
        if (t >= RING) {
          for (;;) {
            const unsigned x = lds_ldv(ctl + ((lane & 8) ? C_INV + t - RING : C_DONE + (lane & 7)));
            const unsigned e = __builtin_amdgcn_readlane(x, 8);
            if (e != 0 && __ballot(x < e) == 0) break;
            __builtin_amdgcn_s_sleep(1);
          }
        }
        MEMBAR();
        LAS u32x4* wd = (LAS u32x4*)(smem + (t >= RING ? t - RING : t) * 8192 + wl);
#pragma unroll
        for (int j = 0; j < 4; j++) wd[j * 64] = o[j];
        MEMBAR();
        lds_st(ctl + h, (d & 1) ? q + d + 1 : q + d + 1 - rx);
        SB();
      }
#pragma unroll
      for (int d = 0; d < NB; d++) so[d] = so[NB + d];
      if (q + NB == 8) z0 = NOW(); if (q + NB == 16) z1 = NOW(); if (q + NB == 24) z2 = NOW();
    }
    if (ETOUCH) {
      auto ro = __builtin_amdgcn_make_buffer_rsrc((void*)pout, 0, MT * N * 4, 0x00020000);
      unsigned sink = 0;
#pragma unroll
      for (int q = 0; q < 2; q++) sink ^= __builtin_amdgcn_raw_buffer_load_b32(ro, (mh * 256 + kq * 64 + (h * 2 + q) * 16 + (lane >> 2)) * (N * 4) + (CMAP ? (nt >> 1) * 1024 + (nt & 1) * 128 + (lane & 3) * 256 : nt * 512 + (lane & 3) * 128), 0, 16);
      if (ETOUCH > 1 && lane < 2) sink ^= __hip_atomic_load((GAS unsigned*)(myws + OFF_CFLAG + (nt * 2 + mh) * 2048 + (kq * 4 + h * 2 + lane) * 128), __ATOMIC_RELAXED, __HIP_MEMORY_SCOPE_AGENT);
      asm volatile("" :: "v"(sink));
    }
    if (h == 0) { STAMP(13); PUT(20, z0); PUT(21, z1); PUT(22, z2); PUT(14, z2); }
    return;
  }
  if (__builtin_expect(wave == 9, 1)) {
    const bool local = un < 32;
    const int sl = local ? un : ((un - 32) * 293) >> 11;
    const int dst = local ? rank : (rank + 1 + (un - 32) - 7 * sl) & 7;
    const unsigned tslot = local ? 0 : SLOT0 + ((unsigned)sl * SLOTLEN >> 3);
    const int r = (rank - dst - 1) & 7;
    const int gs = sl >> 3, gi = (sl >> 1) & 3, smh = sl & 1, skq = (gi + r) & 3;
    const int i = 2 * gi + (r >> 2);
    const int tpos = local ? gs : 4 + 7 * gs + i - (i > 2 * ((skq + 1) & 3) + 1 ? 1 : 0);
    const GAS float* xr = (const GAS float*)px + (size_t)(dst * MT + smh * 256 + (lane & 15)) * KS + (lane >> 4) * 8 + (tpos & 3) * 128 + skq * 32;
    __builtin_amdgcn_s_setprio(SPRIO);
    SB();
    if (!early) {
      __builtin_amdgcn_s_barrier();
      const unsigned tx = min(max(tslot, (unsigned)XLEAD) - XLEAD, (unsigned)XMAX);
      while ((unsigned)__builtin_amdgcn_s_memrealtime() - tstart < tx) __builtin_amdgcn_s_sleep(2);
    }
    MEMBAR();
    SB();
    f32x4 xv[32];
#pragma unroll
    for (int u = 0; u < 16; u++) {
      xv[u * 2] = *(const GAS f32x4*)(xr + u * 16 * KS);
      xv[u * 2 + 1] = *(const GAS f32x4*)(xr + u * 16 * KS + 4);
      SB();
    }
    if (early) __builtin_amdgcn_s_barrier();
    SB();
    const unsigned call = lds_wait_ge(ctl + C_CALL, 1);
    char* dbase = p.ws[dst];
    const unsigned a0 = OFF_A + (call & 1) * A_BYTES + tpos * 131072 + (smh * 4 + skq) * 16384 + lane * 16;
    const unsigned fo = OFF_AFLAG + ((smh * 4 + skq) * 4 + (tpos >> 3)) * 128 + (tpos & 7) * 4;
    u32x4 o[16];
#pragma unroll
    for (int u = 0; u < 16; u++)
      o[u] = u32x4{cvtpk(xv[2 * u][0], xv[2 * u][1]), cvtpk(xv[2 * u][2], xv[2 * u][3]), cvtpk(xv[2 * u + 1][0], xv[2 * u + 1][1]), cvtpk(xv[2 * u + 1][2], xv[2 * u + 1][3])};
    const unsigned long long q0 = NOW();
    while ((unsigned)__builtin_amdgcn_s_memrealtime() - tstart < tslot) __builtin_amdgcn_s_sleep(1);
#pragma unroll
    for (int u = 0; u < 16; u++)
      asm volatile("global_store_dwordx4 %0, %1, %2 offset:%3 sc0 sc1\n\ts_nop 1" :: "v"(a0 + (u >> 2) * 4096), "v"(o[u]), "s"(dbase), "i"((u & 3) * 1024) : "memory");
    if (HOLD == 1) lds_st(ctl + C_SENT, 1);
    asm volatile("s_waitcnt vmcnt(0)" ::: "memory");
    asm volatile("s_mov_b64 exec, 1\n\tglobal_store_dword %0, %1, %2 sc0 sc1\n\ts_mov_b64 exec, -1" :: "v"(fo), "v"(call), "s"(dbase) : "memory");
    if (HOLD == 2) lds_st(ctl + C_SENT, 1);
    STAMP(16); PUT(15, q0);
    return;
  }
  if (wave == 8) {
    __builtin_amdgcn_s_setprio(SPRIO);
    __builtin_amdgcn_s_barrier();
    const unsigned call = sload(myws + OFF_CNT + bid * 4) + 1;
    lds_st(ctl + C_CALL, call);
    if (lane == 0) __hip_atomic_store((GAS unsigned*)(myws + OFF_CNT) + bid, call, __ATOMIC_RELAXED, __HIP_MEMORY_SCOPE_AGENT);
    const unsigned xcc = __builtin_amdgcn_s_getreg((3 << 11) | 20);
    const char* xt = myws + OFF_XTAB + (nt * 2 + mh) * 64;
    if (lane == 0) __hip_atomic_store((GAS unsigned*)xt + kq, call * 16 + xcc, __ATOMIC_RELAXED, __HIP_MEMORY_SCOPE_AGENT);
    const char* af = myws + OFF_AFLAG + (mh * 4 + kq) * 512;
    while ((unsigned)__builtin_amdgcn_s_memrealtime() - tstart < POLL0) __builtin_amdgcn_s_sleep(2);
    int got = 0;
    unsigned todo = 0xffffffffu, here = 0;
    unsigned long long p0 = 0, p1 = 0;
    bool dead = false;
    auto look = [&](int line) {
      const u32x8 f = sload8(af + line * 128);
      unsigned m = 0;
#pragma unroll
      for (int q = 0; q < 8; q++) m |= (f[q] >= call ? 1u : 0u) << q;
      here |= m << (line * 8);
    };
    while (todo) {
      const int low = __builtin_ctz(todo);
      unsigned take = 0;
      if (!dead) look(low >> 3);
      const unsigned missing = todo & ~here;
      take = missing ? todo & ((1u << __builtin_ctz(missing)) - 1) : todo;
      if (DYN && !take && low >= 4) {
        const unsigned x = lds_ldv(ctl + ((lane & 8) ? (lane & 1) : C_DONE + (lane & 7)));
        if (__ballot((lane & 8) == 0 && (int)x + DYN > got) != 0) {
          const int lim = min(min((int)__builtin_amdgcn_readlane(x, 8), (int)__builtin_amdgcn_readlane(x, 9)), low + RING);
          unsigned cand = todo & here;
          if (!cand && low < 24) { look((low >> 3) + 1); cand = todo & here; }
          if (!cand && low < 16) { look((low >> 3) + 2); cand = todo & here; }
          if (cand && __builtin_ctz(cand) < lim) take = cand & -cand;
        }
      }
      if (take) {
        const bool mine = lane < 32 && ((take >> lane) & 1);
        const int e = got + __builtin_popcount(take & ((1u << (lane & 31)) - 1));
        if (mine) { lds_st(ctl + C_ORD + e, lane); lds_st(ctl + C_INV + lane, e + 1); }
        MEMBAR();
        if (got == 0) p0 = NOW();
        const int n = got + __builtin_popcount(take);
        if (got < 11 && n >= 11) p1 = NOW();
        got = n; todo &= ~take;
        lds_st(ctl + C_AST, n == 32 ? 64 : n);
        if (todo && (__builtin_ctz(todo) >> 3) != (low >> 3)) continue;
      } else if (expired(tmo, call, tstart)) { dead = true; here = 0xffffffffu; continue; }
      __builtin_amdgcn_s_sleep(POLL_SLEEP);
    }
    STAMP(19); PUT(17, p0); PUT(18, p1);
    u32x4 xe;
    for (;;) {
      xe = sload4(xt);
      if (((xe[0] >> 4) == call && (xe[1] >> 4) == call && (xe[2] >> 4) == call && (xe[3] >> 4) == call) || expired(tmo, call, tstart)) break;
      __builtin_amdgcn_s_sleep(8);
    }
    lds_st(ctl + C_MODE, (xe[0] == xe[1] && xe[1] == xe[2] && xe[2] == xe[3]) ? 1 : 2);
    return;
  }

  const unsigned call = lds_wait_ge(ctl + C_CALL, 1);
  auto ra = __builtin_amdgcn_make_buffer_rsrc((void*)(myws + OFF_A + (call & 1) * A_BYTES + (mh * 4 + kq) * 16384), 0, AREC ? 31 * 131072 + 16384 : 0, 0x00020000);
  const int r0 = (1 + (kq >> 1)) & 1;
  const int vl = lane * 16;
  const int y = wave >> 2, rw2 = YS ? (wave & 3) * 2 + ((y + r0) & 1) : wave;
  const int va0 = vl + rw2 * 2048 + (YS ? 0 : r0 * 1024), va1 = va0 ^ 1024;
  const int lag = y * LAG;
  f32x4 acc[2][8];
#pragma unroll
  for (int i = 0; i < 2; i++)
#pragma unroll
    for (int q = 0; q < 8; q++) { acc[i][q] = f32x4{0, 0, 0, 0}; asm volatile("" : "+v"(acc[i][q])); }
  u32x4 ab[4][2], bf[8];
  int wlim = 0;
  int alim = 0;
  auto wgate = [&](int t) {
    if (t >= wlim) {
      do {
        const int w = min(lds_ld(ctl + C_W0), lds_ld(ctl + C_W1));
        wlim = (w >= 32) ? 64 : w - lag;
        if (GSLEEP && t >= wlim) __builtin_amdgcn_s_sleep(GSLEEP);
      } while (t >= wlim);
    }
    MEMBAR();
  };
  int ord = 0;
  auto agate = [&](int e) {
    if (e >= alim) { alim = lds_wait_ge(ctl + C_AST, e + 1); ord = lds_ldv(ctl + C_ORD + lane); asm volatile("" : "+v"(ord)); }
    MEMBAR();
    return __builtin_amdgcn_readlane(ord, e);
  };
  wgate(0);
#pragma unroll
  for (int q = 0; q < 8; q++) { bf[q] = *(LAS const u32x4*)(smem + vl + q * 1024); SB(); }
  agate(ALOOK - 1);
  const unsigned long long sb = NOW();
  unsigned long long s8 = 0, s16 = 0, s24 = 0;
#pragma unroll
  for (int d = 0; d < ALOOK; d++) {
    ab[d][0] = __builtin_amdgcn_raw_buffer_load_b128(ra, va0, d * 131072, 17); SB();
    ab[d][1] = __builtin_amdgcn_raw_buffer_load_b128(ra, va1, d * 131072, 17); SB();
  }
#pragma nounroll
  for (int t0 = 0; t0 < 32; t0 += 4) {
    if (BALP == 1 && y && t0 == BALT) __builtin_amdgcn_s_setprio(1);
    if (BALP == 2 && y) { if (t0 & 4) __builtin_amdgcn_s_setprio(1); else __builtin_amdgcn_s_setprio(0); }
#pragma unroll
    for (int u = 0; u < 4; u++) {
      const int t = t0 + u;
      if (BALP == 4 && y) { if (u & 1) __builtin_amdgcn_s_setprio(1); else __builtin_amdgcn_s_setprio(0); }
      if (BALD && (u & 1) == 0 && t0 >= 8) {
        while (__ballot((int)lds_ldv(ctl + C_DONE + (lane & 7)) + BALD < t + 1) != 0) __builtin_amdgcn_s_sleep(1);
        MEMBAR();
      }
      const int pa = agate(t + ALOOK);
      SB();
      ab[(u + ALOOK) & 3][0] = __builtin_amdgcn_raw_buffer_load_b128(ra, va0, pa * 131072, 17); SB();
      ab[(u + ALOOK) & 3][1] = __builtin_amdgcn_raw_buffer_load_b128(ra, va1, pa * 131072, 17); SB();
      const int pw = __builtin_amdgcn_readlane(ord, t + 1);
      wgate(pw);
      LAS const u32x4* wp = (LAS const u32x4*)(smem + vl + (pw >= RING ? pw - RING : pw) * 8192);
      SB();
#pragma unroll
      for (int q = 0; q < 8; q++) {
        MFMA(acc[0][q], ab[u][0], bf[q]);
        MFMA(acc[1][q], ab[u][1], bf[q]);
        SB();
        bf[q] = wp[q * 64];
        SB();
      }
      MEMBAR();
      lds_st(ctl + C_DONE + wave, t + 2);
      SB();
    }
    if (t0 == 4) s8 = NOW(); if (t0 == 12) s16 = NOW(); if (t0 == 20) s24 = NOW();
  }
  const unsigned long long se = NOW();
  const unsigned mode = lds_wait_ge(ctl + C_MODE, 1);
  auto rc = __builtin_amdgcn_make_buffer_rsrc((void*)(myws + OFF_CFLAG), 0, 131072, 0x00020000);
  const int vo = (mh * 256 + rw2 * 32 + (lane >> 4) * 4) * (N * 4) + colb;
  const int fo = ((nt * 2 + mh) * 8 + rw2) * 256;
  if (RPRIO) __builtin_amdgcn_s_setprio(RPRIO);
  asm volatile("s_nop 7");
  SB();
  if (YS) {
    if (mode == 1) ring_reduce2<0, RLDA>(acc, pout, vo, kq, y, rc, fo, call, tmo, lane, tstart);
    else ring_reduce2<16, 16>(acc, pout, vo, kq, y, rc, fo, call, tmo, lane, tstart);
  } else {
    if (mode == 1) ring_reduce<0, RLDA>(acc, pout, vo, kq, rc, fo, call, tmo, lane, tstart);
    else ring_reduce<16, 16>(acc, pout, vo, kq, rc, fo, call, tmo, lane, tstart);
  }
  if (WBL == 1 && wave == 0) asm volatile("s_waitcnt vmcnt(0)\n\tbuffer_wbl2 sc1" ::: "memory");
  if (WBL == 2 && wave == 0 && (idx & 7) == 0) asm volatile("s_waitcnt vmcnt(0)\n\tbuffer_wbl2 sc1" ::: "memory");
}

extern "C" void kernel_launch(int rank, int nranks, ncclComm_t comm, hipStream_t stream,
                              void* const* d_in, const int* in_sizes, int n_in,
                              void* d_out, int out_size, const int* in_dtypes, int out_dtype,
                              void* const* d_out_all, void* const* d_ws_all, size_t ws_bytes) {
  if (nranks != NR || (unsigned)rank >= (unsigned)NR || ws_bytes < WS_NEED) { fprintf(stderr, "bad config\n"); return; }
  static bool init = false;
  if (!init) { HIP_CHECK(hipFuncSetAttribute((const void*)a2a_gemm, hipFuncAttributeMaxDynamicSharedMemorySize, LDS_BYTES)); init = true; }
  Params p;
  p.x = (const float*)d_in[0]; p.w = (const float*)d_in[1]; p.out = (float*)d_out; p.rank = rank; p.pad = 0; p.myws = (char*)d_ws_all[rank];
  for (int i = 0; i < NR; i++) p.ws[i] = (char*)d_ws_all[i];
  hipLaunchKernelGGL(a2a_gemm, dim3(256), dim3(768), LDS_BYTES, stream, p);
  HIP_CHECK(hipGetLastError());
}
```

```cpp
#include <hip/hip_runtime.h>
#include <rccl/rccl.h>
#include <cstddef>
#include <cstdio>

#define HIP_CHECK(expr) do { hipError_t _e = (expr); if (_e != hipSuccess) { \
    fprintf(stderr, "HIP error %s at %s:%d\n", hipGetErrorString(_e), __FILE__, __LINE__); \
    return; } } while (0)

typedef unsigned u32x4 __attribute__((ext_vector_type(4)));
typedef unsigned u32x8 __attribute__((ext_vector_type(8)));
typedef float f32x4 __attribute__((ext_vector_type(4)));
#define LAS __attribute__((address_space(3)))
#define GAS __attribute__((address_space(1)))
#define SB() __builtin_amdgcn_sched_barrier(0)
#define MEMBAR() asm volatile("" ::: "memory")

constexpr int NR = 8, MT = 512, N = 4096, K = 4096, KS = 512;
constexpr int ROW = N * 4;
constexpr unsigned A_BYTES = MT * K * 2;
constexpr unsigned OFF_A = 0;
constexpr unsigned OFF_AFLAG = 2 * A_BYTES;
constexpr unsigned OFF_CFLAG = OFF_AFLAG + 4096;
constexpr unsigned OFF_XTAB = OFF_CFLAG + 131072;
constexpr unsigned OFF_TMO = OFF_XTAB + 4096;
constexpr unsigned OFF_CNT = OFF_TMO + 4096;
constexpr unsigned WS_NEED = OFF_CNT + 4096;
constexpr unsigned DEADLINE = 50000000u;

constexpr int RING = 19;
constexpr unsigned SLOT0 = 150;
constexpr unsigned SLOTLEN = 240;
constexpr unsigned XAT = 100;
constexpr int EARLY = 128;
constexpr unsigned POLL0 = 300;
constexpr int DYN = 4;
constexpr unsigned CPF = 10944;
constexpr int PSLEEP = 3;
constexpr int HOLD = 32;

constexpr int LDS_CTL = RING * 8192, LDS_BYTES = LDS_CTL + 512;
static_assert(LDS_BYTES <= 163840 && RING >= 16, "LDS");
enum { C_W0 = 0, C_W1 = 1, C_AST = 2, C_MODE = 3, C_CALL = 4, C_SENT = 5, C_DONE = 8, C_INV = 16, C_ORD = 64 };

struct Params { const float* x; const float* w; float* out; char* myws; int rank; int pad; char* ws[NR]; };
static_assert(offsetof(Params, w) == 8 && offsetof(Params, out) == 16 && offsetof(Params, myws) == 24 && offsetof(Params, rank) == 32, "kernarg");

__device__ __forceinline__ unsigned cvtpk(float a, float b) {
  unsigned r; asm volatile("v_cvt_pk_f16_f32 %0, %1, %2" : "=v"(r) : "v"(a), "v"(b)); return r;
}
__device__ __forceinline__ unsigned ticks() { return (unsigned)__builtin_amdgcn_s_memrealtime(); }
__device__ __forceinline__ unsigned lds_ldv(LAS unsigned* p) { return __hip_atomic_load(p, __ATOMIC_RELAXED, __HIP_MEMORY_SCOPE_WORKGROUP); }
__device__ __forceinline__ unsigned lds_ld(LAS unsigned* p) { return __builtin_amdgcn_readfirstlane(lds_ldv(p)); }
__device__ __forceinline__ void lds_st(LAS unsigned* p, unsigned v) { __hip_atomic_store(p, v, __ATOMIC_RELAXED, __HIP_MEMORY_SCOPE_WORKGROUP); }
__device__ __forceinline__ unsigned lds_wait_ge(LAS unsigned* p, unsigned v) {
  unsigned r;
  while ((r = lds_ld(p)) < v) __builtin_amdgcn_s_sleep(1);
  MEMBAR();
  return r;
}
__device__ __forceinline__ bool expired(GAS unsigned* tmo, unsigned call, unsigned t0) {
  const unsigned dt = ticks() - t0;
  if (dt < 20000) return false;
  if (dt > DEADLINE || __builtin_amdgcn_readfirstlane(__hip_atomic_load(tmo, __ATOMIC_RELAXED, __HIP_MEMORY_SCOPE_AGENT)) == call) {
    __hip_atomic_store(tmo, call, __ATOMIC_RELAXED, __HIP_MEMORY_SCOPE_AGENT);
    asm volatile("s_waitcnt vmcnt(0)" ::: "memory");
    return true;
  }
  return false;
}
__device__ __forceinline__ unsigned sload(const void* p) { unsigned r; asm volatile("s_load_dword %0, %1, 0x0 glc\n\ts_waitcnt lgkmcnt(0)" : "=s"(r) : "s"(p) : "memory"); return r; }
__device__ __forceinline__ u32x8 sload8(const void* p) { u32x8 r; asm volatile("s_load_dwordx8 %0, %1, 0x0 glc\n\ts_waitcnt lgkmcnt(0)" : "=s"(r) : "s"(p) : "memory"); return r; }
#define MFMA(acc, a, b) asm volatile("v_mfma_f32_16x16x32_f16 %0, %1, %2, %0" : "+v"(acc) : "v"(a), "v"(b))

template <int STA>
__device__ __forceinline__ void ring_reduce(f32x4 (&acc)[2][8], float* out, int vo, int kq, __amdgpu_buffer_rsrc_t rc, int fo,
                                            unsigned call, GAS unsigned* tmo, int lane, unsigned t0) {
  auto ro = __builtin_amdgcn_make_buffer_rsrc((void*)out, 0, MT * ROW, 0x00020000);
#pragma unroll
  for (int v = 0; v < 4; v++) {
    const int i = v >> 1, c0 = (v & 1) * 4;
    const int rg = (i + 1 + (kq >> 1)) & 1, hf = (v ^ kq) & 1;
    const int so = rg * 16 * ROW + hf * 256, fv = fo + (rg * 2 + hf) * 64;
    f32x4 o[4];
    if (v > 0) {
      int spins = 0;
      while (__builtin_amdgcn_readfirstlane(__builtin_amdgcn_raw_buffer_load_b32(rc, 0, fv, 16)) < call * 4 + v) {
        MEMBAR();
        if (++spins > 64) { if (expired(tmo, call, t0)) break; __builtin_amdgcn_s_sleep(2); }
      }
      MEMBAR();
#pragma unroll
      for (int e = 0; e < 4; e++) { o[e] = __builtin_bit_cast(f32x4, __builtin_amdgcn_raw_buffer_load_b128(ro, vo, so + e * ROW, 16)); SB(); }
    }
    if (v == 3 && __hip_atomic_load(tmo, __ATOMIC_RELAXED, __HIP_MEMORY_SCOPE_AGENT) == call) {
      const float q = __builtin_nanf("");
#pragma unroll
      for (int e = 0; e < 4; e++) o[e] = f32x4{q, q, q, q};
    }
#pragma unroll
    for (int e = 0; e < 4; e++) {
#pragma unroll
      for (int c = 0; c < 4; c++) {
        float t_;
        if (v > 0) { t_ = o[e][c]; asm volatile("v_add_f32 %0, %0, %1" : "+v"(t_) : "v"(acc[i][c0 + c][e])); }
        else t_ = acc[i][c0 + c][e];
        o[e][c] = t_;
      }
      asm volatile("" : "+v"(o[e]));
      __builtin_amdgcn_raw_buffer_store_b128(__builtin_bit_cast(u32x4, o[e]), ro, vo, so + e * ROW, STA);
      SB();
      asm volatile("s_nop 1");
      SB();
    }
    if (v < 3) {
      asm volatile("s_waitcnt vmcnt(0)" ::: "memory");
      if (lane == 0) __builtin_amdgcn_raw_buffer_store_b32(call * 4 + v + 1, rc, 0, fv, STA);
    }
    SB();
  }
}

__global__ __launch_bounds__(768, 3) void a2a_gemm(Params p) {
  extern __shared__ __attribute__((aligned(16))) char smem_[];
  LAS char* smem = (LAS char*)smem_;
  LAS unsigned* ctl = (LAS unsigned*)(smem + LDS_CTL);
  u32x8 ka; unsigned karank; unsigned long long tclk;
  asm volatile("s_load_dwordx8 %0, %3, 0x0\n\ts_load_dword %1, %3, 0x20\n\ts_memrealtime %2"
               : "=&s"(ka), "=&s"(karank), "=&s"(tclk) : "s"(__builtin_amdgcn_kernarg_segment_ptr()));
  const int lane = threadIdx.x & 63;
  const int wave = __builtin_amdgcn_readfirstlane(threadIdx.x >> 6);
  if (wave == 0) {
    lds_st(ctl + lane, 0); lds_st(ctl + C_ORD + lane, 32);
    asm volatile("s_waitcnt lgkmcnt(0)\n\ts_barrier" ::: "memory");
  } else if (wave < 8) {
    unsigned long long pc; unsigned junk;
    const unsigned off = min(((blockIdx.x >> 3) * 7 + wave - 1) * 64u, CPF - 64u);
    asm volatile("s_getpc_b64 %0\n\ts_load_dword %1, %0, %2\n\ts_barrier\n\ts_waitcnt lgkmcnt(0)" : "=&s"(pc), "=&s"(junk) : "s"(off) : "memory");
  }
  const int bid = blockIdx.x;
  const int idx = bid >> 3;
  const int kq = idx & 3, mh = (idx >> 2) & 1, ntl = idx >> 3, nt = (bid & 7) * 4 + ntl;
  const int un = (~ntl & 1) * 128 + ((mh ^ (ntl >> 1)) & 1) * 64 + (kq & 1) * 32 + (kq >> 1) * 16 + (ntl >> 1) * 8 + (bid & 7);
  const bool early = un < EARLY;
  int vw = ((lane >> 5) * 8 + (wave - 10) * 16 + kq * 32) * ROW + nt * 512 + (lane & 31) * 16;
  asm volatile("s_waitcnt lgkmcnt(0)" : "+s"(ka), "+s"(karank), "+s"(tclk), "+v"(vw));
  const float* px = (const float*)(((unsigned long long)ka[1] << 32) | ka[0]);
  const float* pw = (const float*)(((unsigned long long)ka[3] << 32) | ka[2]);
  float* pout = (float*)(((unsigned long long)ka[5] << 32) | ka[4]);
  char* myws = (char*)(((unsigned long long)ka[7] << 32) | ka[6]);
  const int rank = karank;
  const unsigned tstart = (unsigned)tclk;
  GAS unsigned* tmo = (GAS unsigned*)(myws + OFF_TMO);


  if (__builtin_expect(wave >= 10, 1)) {
    const int h = wave - 10;
    auto rw = __builtin_amdgcn_make_buffer_rsrc((void*)pw, 0, K * ROW, 0x00020000);
    if (early) __builtin_amdgcn_s_barrier();
    if (un < HOLD) lds_wait_ge(ctl + C_SENT, 1);
    MEMBAR();
    u32x4 wb[16];
#pragma unroll
    for (int n = 0; n < 16; n++) { wb[n] = __builtin_amdgcn_raw_buffer_load_b128(rw, vw, (rank * 4 + (n >> 3)) * (128 * ROW) + (n & 7) * ROW, 0); SB(); }
    if (!early) __builtin_amdgcn_s_barrier();
    MEMBAR();
    const int wl = (((lane >> 4) ^ kq) & 1) * 4096 + (h * 2 + (lane >> 5)) * 256 + (lane & 15) * 16;
    const int miss = 2 * ((kq + 1) & 3) + 1;
#pragma nounroll
    for (int q = 0; q < 32; q += 2) {
      const int ve = (q < 30) ? vw : 0x40000000;
#pragma unroll
      for (int d = 0; d < 2; d++) {
        const int t = q + d, tn = (t + 2) & 31;
        const int u = tn - 4, j7 = u - 7 * ((u * 37) >> 8), i8 = j7 + (j7 >= miss ? 1 : 0);
        int src; asm("s_cmp_lt_i32 %1, 4\n\ts_cselect_b32 %0, %2, %3" : "=s"(src) : "s"(tn), "s"(rank), "s"(rank + 1 + ((kq - (i8 >> 1)) & 3) + 4 * (i8 & 1)) : "scc");
        const int so = ((src & 7) * 4 + (tn & 3)) * (128 * ROW);
        u32x4 o[4];
#pragma unroll
        for (int i = 0; i < 4; i++) {
          const int m = d * 8 + 2 * i;
          const f32x4 w0 = __builtin_bit_cast(f32x4, wb[m]), w1 = __builtin_bit_cast(f32x4, wb[m + 1]);
#pragma unroll
          for (int j = 0; j < 4; j++) o[j][i] = cvtpk(w0[j], w1[j]);
          SB();
          wb[m] = __builtin_amdgcn_raw_buffer_load_b128(rw, ve, so + (2 * i) * ROW, 0); SB();
          wb[m + 1] = __builtin_amdgcn_raw_buffer_load_b128(rw, ve, so + (2 * i + 1) * ROW, 0); SB();
        }
        if (t >= RING) {
          for (;;) {
            const unsigned x = lds_ldv(ctl + ((lane & 8) ? C_INV + t - RING : C_DONE + (lane & 7)));
            const unsigned e = __builtin_amdgcn_readlane(x, 8);
            if (e != 0 && __ballot(x < e) == 0) break;
            __builtin_amdgcn_s_sleep(1);
          }
        }
        MEMBAR();
        LAS u32x4* wd = (LAS u32x4*)(smem + (t >= RING ? t - RING : t) * 8192 + wl);
#pragma unroll
        for (int j = 0; j < 4; j++) wd[j * 64] = o[j];
        MEMBAR();
        lds_st(ctl + h, t + 1);
        SB();
      }
    }
    {
      auto ro = __builtin_amdgcn_make_buffer_rsrc((void*)pout, 0, MT * ROW, 0x00020000);
      unsigned sink = 0;
#pragma unroll
      for (int q = 0; q < 2; q++)
        sink ^= __builtin_amdgcn_raw_buffer_load_b32(ro, (mh * 256 + kq * 64 + (h * 2 + q) * 16 + (lane >> 2)) * ROW + nt * 512 + (lane & 3) * 128, 0, 16);
      if (lane < 3) sink ^= __hip_atomic_load((GAS unsigned*)(myws + (lane == 2 ? OFF_TMO : OFF_CFLAG + (nt * 2 + mh) * 2048 + (kq * 4 + h * 2 + lane) * 128)), __ATOMIC_RELAXED, __HIP_MEMORY_SCOPE_AGENT);
      asm volatile("" :: "v"(sink));
    }
    return;
  }

  if (__builtin_expect(wave == 9, 1)) {
    const bool local = un < 32;
    const int sl = local ? un : ((un - 32) * 293) >> 11;
    const int dst = (local ? rank : rank + 1 + (un - 32) - 7 * sl) & 7;
    const unsigned tslot = local ? 0 : SLOT0 + ((unsigned)sl * SLOTLEN >> 3);
    const int r = (rank - dst - 1) & 7;
    const int gs = sl >> 3, gi = (sl >> 1) & 3, smh = sl & 1, skq = (gi + r) & 3;
    const int i8 = 2 * gi + (r >> 2);
    const int tpos = local ? gs : 4 + 7 * gs + i8 - (i8 > 2 * ((skq + 1) & 3) + 1 ? 1 : 0);
    const GAS float* xr = (const GAS float*)px + (size_t)(dst * MT + smh * 256 + (lane & 15)) * KS + (lane >> 4) * 8 + (tpos & 3) * 128 + skq * 32;
    __builtin_amdgcn_s_setprio(3);
    SB();
    if (!early) {
      __builtin_amdgcn_s_barrier();
      while (ticks() - tstart < XAT) __builtin_amdgcn_s_sleep(2);
    }
    MEMBAR();
    SB();
    f32x4 xv[32];
#pragma unroll
    for (int u = 0; u < 16; u++) {
      xv[u * 2] = *(const GAS f32x4*)(xr + u * 16 * KS);
      xv[u * 2 + 1] = *(const GAS f32x4*)(xr + u * 16 * KS + 4);
      SB();
    }
    if (early) __builtin_amdgcn_s_barrier();
    SB();
    const unsigned call = lds_wait_ge(ctl + C_CALL, 1);
    char* dbase = p.ws[dst];
    const unsigned a0 = OFF_A + (call & 1) * A_BYTES + tpos * 131072 + (smh * 4 + skq) * 16384 + lane * 16;
    const unsigned fo = OFF_AFLAG + ((smh * 4 + skq) * 4 + (tpos >> 3)) * 128 + (tpos & 7) * 4;
    u32x4 o[16];
#pragma unroll
    for (int u = 0; u < 16; u++)
      o[u] = u32x4{cvtpk(xv[2 * u][0], xv[2 * u][1]), cvtpk(xv[2 * u][2], xv[2 * u][3]), cvtpk(xv[2 * u + 1][0], xv[2 * u + 1][1]), cvtpk(xv[2 * u + 1][2], xv[2 * u + 1][3])};
    while (ticks() - tstart < tslot) __builtin_amdgcn_s_sleep(1);
#pragma unroll
    for (int u = 0; u < 16; u++)
      asm volatile("global_store_dwordx4 %0, %1, %2 offset:%3 sc0 sc1\n\ts_nop 1" :: "v"(a0 + (u >> 2) * 4096), "v"(o[u]), "s"(dbase), "i"((u & 3) * 1024) : "memory");
    asm volatile("s_waitcnt vmcnt(0)" ::: "memory");
    asm volatile("s_mov_b64 exec, 1\n\tglobal_store_dword %0, %1, %2 sc0 sc1\n\ts_mov_b64 exec, -1" :: "v"(fo), "v"(call), "s"(dbase) : "memory");
    lds_st(ctl + C_SENT, 1);
    GAS unsigned* xt = (GAS unsigned*)(myws + OFF_XTAB + (nt * 2 + mh) * 64);
    unsigned xe;
    for (;;) {
      xe = __hip_atomic_load(xt + (lane & 3), __ATOMIC_RELAXED, __HIP_MEMORY_SCOPE_AGENT);
      if (__ballot((xe >> 4) != call) == 0 || expired(tmo, call, tstart)) break;
      __builtin_amdgcn_s_sleep(8);
    }
    lds_st(ctl + C_MODE, __ballot(xe != (unsigned)__builtin_amdgcn_readfirstlane(xe)) == 0 ? 1 : 2);
    return;
  }

  if (wave == 8) {
    __builtin_amdgcn_s_setprio(3);
    __builtin_amdgcn_s_barrier();
    const unsigned call = sload(myws + OFF_CNT + bid * 4) + 1;
    lds_st(ctl + C_CALL, call);
    const unsigned xcc = __builtin_amdgcn_s_getreg((3 << 11) | 20);
    GAS unsigned* xt = (GAS unsigned*)(myws + OFF_XTAB + (nt * 2 + mh) * 64);
    if (lane == 0) {
      __hip_atomic_store((GAS unsigned*)(myws + OFF_CNT) + bid, call, __ATOMIC_RELAXED, __HIP_MEMORY_SCOPE_AGENT);
      __hip_atomic_store(xt + kq, call * 16 + xcc, __ATOMIC_RELAXED, __HIP_MEMORY_SCOPE_AGENT);
    }
    const char* af = myws + OFF_AFLAG + (mh * 4 + kq) * 512;
    while (ticks() - tstart < POLL0) __builtin_amdgcn_s_sleep(2);
    int got = 0;
    unsigned todo = 0xffffffffu, here = 0;
    auto look = [&](int line) {
      const u32x8 f = sload8(af + line * 128);
      unsigned m = 0;
#pragma unroll
      for (int q = 0; q < 8; q++) m |= (f[q] >= call ? 1u : 0u) << q;
      here |= m << (line * 8);
    };
    while (todo) {
      const int low = __builtin_ctz(todo);
      look(low >> 3);
      const unsigned missing = todo & ~here;
      unsigned take = missing ? todo & ((1u << __builtin_ctz(missing)) - 1) : todo;
      if (!take && low >= 4) {
        const unsigned x = lds_ldv(ctl + ((lane & 8) ? (lane & 1) : C_DONE + (lane & 7)));
        if (__ballot((lane & 8) == 0 && (int)x + DYN > got) != 0) {
          const int lim = min((int)__builtin_amdgcn_readlane(x, 8), (int)__builtin_amdgcn_readlane(x, 9));
          unsigned cand = todo & here;
          if (!cand && low < 24) { look((low >> 3) + 1); cand = todo & here; }
          if (!cand && low < 16) { look((low >> 3) + 2); cand = todo & here; }
          if (cand && __builtin_ctz(cand) < lim) take = cand & -cand;
        }
      }
      if (take) {
        const int e = got + __builtin_popcount(take & ((1u << (lane & 31)) - 1));
        if (lane < 32 && ((take >> lane) & 1)) { lds_st(ctl + C_ORD + e, lane); lds_st(ctl + C_INV + lane, e + 1); }
        MEMBAR();
        const int n = got + __builtin_popcount(take);
        got = n; todo &= ~take;
        lds_st(ctl + C_AST, n == 32 ? 64 : n);
        if (todo && (__builtin_ctz(todo) >> 3) != (low >> 3)) continue;
      } else if (ticks() - tstart > 5000) {
        const unsigned f = __hip_atomic_load((const GAS unsigned*)(af + ((lane >> 3) & 3) * 128) + (lane & 7), __ATOMIC_RELAXED, __HIP_MEMORY_SCOPE_SYSTEM);
        here |= (unsigned)__ballot(f >= call);
        if (expired(tmo, call, tstart)) here = 0xffffffffu;
      }
      __builtin_amdgcn_s_sleep(PSLEEP);
    }
    return;
  }

  const unsigned call = lds_wait_ge(ctl + C_CALL, 1);
  auto ra = __builtin_amdgcn_make_buffer_rsrc((void*)(myws + OFF_A + (call & 1) * A_BYTES + (mh * 4 + kq) * 16384), 0, 31 * 131072 + 16384, 0x00020000);
  const int vl = lane * 16;
  const int va0 = vl + wave * 2048 + ((1 + (kq >> 1)) & 1) * 1024, va1 = va0 ^ 1024;
  f32x4 acc[2][8];
#pragma unroll
  for (int i = 0; i < 2; i++)
#pragma unroll
    for (int q = 0; q < 8; q++) { acc[i][q] = f32x4{0, 0, 0, 0}; asm volatile("" : "+v"(acc[i][q])); }
  u32x4 ab[4][2], bf[8];
  int wlim = 0;
  int alim = 0;
  int ord = 0;
  auto wgate = [&](int t) {
    if (t >= wlim) {
      do {
        const int w = min(lds_ld(ctl + C_W0), lds_ld(ctl + C_W1));
        wlim = (w >= 32) ? 64 : w;
        if (t >= wlim) __builtin_amdgcn_s_sleep(1);
      } while (t >= wlim);
    }
    MEMBAR();
  };
  auto agate = [&](int e) {
    if (e >= alim) { alim = lds_wait_ge(ctl + C_AST, e + 1); ord = lds_ldv(ctl + C_ORD + lane); asm volatile("" : "+v"(ord)); }
    MEMBAR();
    return __builtin_amdgcn_readlane(ord, e);
  };
  wgate(0);
#pragma unroll
  for (int q = 0; q < 8; q++) { bf[q] = *(LAS const u32x4*)(smem + vl + q * 1024); SB(); }
  agate(1);
#pragma unroll
  for (int d = 0; d < 2; d++) {
    ab[d][0] = __builtin_amdgcn_raw_buffer_load_b128(ra, va0, d * 131072, 17); SB();
    ab[d][1] = __builtin_amdgcn_raw_buffer_load_b128(ra, va1, d * 131072, 17); SB();
  }
  unsigned mode = 0;
#pragma nounroll
  for (int t0 = 0; t0 < 32; t0 += 4) {
#pragma unroll
    for (int u = 0; u < 4; u++) {
      const int t = t0 + u;
      if (u == 2 && t0 == 28) mode = lds_ld(ctl + C_MODE);
      const int pa = agate(t + 2);
      SB();
      ab[(u + 2) & 3][0] = __builtin_amdgcn_raw_buffer_load_b128(ra, va0, pa * 131072, 17); SB();
      ab[(u + 2) & 3][1] = __builtin_amdgcn_raw_buffer_load_b128(ra, va1, pa * 131072, 17); SB();
      const int pn = __builtin_amdgcn_readlane(ord, t + 1);
      wgate(pn);
      LAS const u32x4* wp = (LAS const u32x4*)(smem + vl + (pn >= RING ? pn - RING : pn) * 8192);
      SB();
#pragma unroll
      for (int q = 0; q < 8; q++) {
        MFMA(acc[0][q], ab[u][0], bf[q]);
        MFMA(acc[1][q], ab[u][1], bf[q]);
        SB();
        bf[q] = wp[q * 64];
        SB();
      }
      MEMBAR();
      lds_st(ctl + C_DONE + wave, t + 2);
      SB();
    }
  }
  if (mode == 0) mode = lds_wait_ge(ctl + C_MODE, 1);
  auto rc = __builtin_amdgcn_make_buffer_rsrc((void*)(myws + OFF_CFLAG), 0, 131072, 0x00020000);
  const int vo = (mh * 256 + wave * 32 + (lane >> 4) * 4) * ROW + nt * 512 + (lane & 15) * 16;
  const int fo = ((nt * 2 + mh) * 8 + wave) * 256;
  asm volatile("s_nop 7"
               : "+v"(acc[0][0]), "+v"(acc[0][1]), "+v"(acc[0][2]), "+v"(acc[0][3]), "+v"(acc[0][4]), "+v"(acc[0][5]), "+v"(acc[0][6]), "+v"(acc[0][7]),
                 "+v"(acc[1][0]), "+v"(acc[1][1]), "+v"(acc[1][2]), "+v"(acc[1][3]), "+v"(acc[1][4]), "+v"(acc[1][5]), "+v"(acc[1][6]), "+v"(acc[1][7]));
  SB();
  if (__builtin_expect(mode == 1, 1)) ring_reduce<0>(acc, pout, vo, kq, rc, fo, call, tmo, lane, tstart);
  else ring_reduce<16>(acc, pout, vo, kq, rc, fo, call, tmo, lane, tstart);
}

extern "C" void kernel_launch(int rank, int nranks, ncclComm_t comm, hipStream_t stream,
                              void* const* d_in, const int* in_sizes, int n_in,
                              void* d_out, int out_size, const int* in_dtypes, int out_dtype,
                              void* const* d_out_all, void* const* d_ws_all, size_t ws_bytes) {
  if (nranks != NR || (unsigned)rank >= (unsigned)NR || ws_bytes < WS_NEED || n_in != 2 || in_sizes[0] != NR * MT * KS ||
      in_sizes[1] != K * N || out_size != MT * N) { fprintf(stderr, "unexpected configuration\n"); return; }
  static bool init = false;
  if (!init) { HIP_CHECK(hipFuncSetAttribute((const void*)a2a_gemm, hipFuncAttributeMaxDynamicSharedMemorySize, LDS_BYTES)); init = true; }
  Params p;
  p.x = (const float*)d_in[0]; p.w = (const float*)d_in[1]; p.out = (float*)d_out;
  p.myws = (char*)d_ws_all[rank]; p.rank = rank; p.pad = 0;
  for (int i = 0; i < NR; i++) p.ws[i] = (char*)d_ws_all[i];
  hipLaunchKernelGGL(a2a_gemm, dim3(256), dim3(768), LDS_BYTES, stream, p);
  HIP_CHECK(hipGetLastError());
}
```

```cpp
#include <hip/hip_runtime.h>
#include <rccl/rccl.h>
#include <cstddef>
#include <cstdio>

#define HIP_CHECK(expr) do { hipError_t _e = (expr); if (_e != hipSuccess) { \
    fprintf(stderr, "HIP error %s at %s:%d\n", hipGetErrorString(_e), __FILE__, __LINE__); \
    return; } } while (0)

typedef unsigned u32x4 __attribute__((ext_vector_type(4)));
typedef unsigned u32x8 __attribute__((ext_vector_type(8)));
typedef float f32x4 __attribute__((ext_vector_type(4)));
#define LAS __attribute__((address_space(3)))
#define GAS __attribute__((address_space(1)))
#define SB() __builtin_amdgcn_sched_barrier(0)
#define MEMBAR() asm volatile("" ::: "memory")

constexpr int NR = 8, MT = 512, N = 4096, K = 4096, KS = 512;
constexpr int ROW = N * 4;
constexpr unsigned A_BYTES = MT * K * 2;
constexpr unsigned OFF_A = 0;
constexpr unsigned OFF_AFLAG = 2 * A_BYTES;
constexpr unsigned OFF_CFLAG = OFF_AFLAG + 16384;
constexpr unsigned OFF_XTAB = OFF_CFLAG + 131072;
constexpr unsigned OFF_TMO = OFF_XTAB + 4096;
constexpr unsigned OFF_CNT = OFF_TMO + 4096;
constexpr unsigned WS_NEED = OFF_CNT + 32768;
constexpr unsigned DEADLINE = 50000000u;

constexpr int RING = 19;
constexpr unsigned SLOT0 = 150;
constexpr unsigned SLOTLEN = 240;
constexpr unsigned XAT = 100;
constexpr int EARLY = 128;
constexpr unsigned POLL0 = 300;
constexpr int DYN = 4;
constexpr unsigned CPF = 10496;
constexpr int PSLEEP = 3;
constexpr int HOLD = 32;

constexpr int LDS_CTL = RING * 8192, LDS_BYTES = LDS_CTL + 512;
static_assert(LDS_BYTES <= 163840 && RING >= 16, "LDS");
enum { C_W0 = 0, C_W1 = 1, C_AST = 2, C_MODE = 3, C_CALL = 4, C_SENT = 5, C_DONE = 8, C_INV = 16, C_ORD = 64 };

struct Params { const float* x; const float* w; float* out; char* myws; int rank; int pad; char* ws[NR]; };
static_assert(offsetof(Params, w) == 8 && offsetof(Params, out) == 16 && offsetof(Params, myws) == 24 && offsetof(Params, rank) == 32, "kernarg");

__device__ __forceinline__ unsigned cvtpk(float a, float b) {
  unsigned r; asm volatile("v_cvt_pk_f16_f32 %0, %1, %2" : "=v"(r) : "v"(a), "v"(b)); return r;
}
__device__ __forceinline__ unsigned ticks() { return (unsigned)__builtin_amdgcn_s_memrealtime(); }
__device__ __forceinline__ unsigned lds_ldv(LAS unsigned* p) { return __hip_atomic_load(p, __ATOMIC_RELAXED, __HIP_MEMORY_SCOPE_WORKGROUP); }
__device__ __forceinline__ unsigned lds_ld(LAS unsigned* p) { return __builtin_amdgcn_readfirstlane(lds_ldv(p)); }
__device__ __forceinline__ void lds_st(LAS unsigned* p, unsigned v) { __hip_atomic_store(p, v, __ATOMIC_RELAXED, __HIP_MEMORY_SCOPE_WORKGROUP); }
__device__ __forceinline__ unsigned lds_wait_ge(LAS unsigned* p, unsigned v) {
  unsigned r;
  while ((r = lds_ld(p)) < v) __builtin_amdgcn_s_sleep(1);
  MEMBAR();
  return r;
}
__device__ __forceinline__ bool expired(GAS unsigned* tmo, unsigned call, unsigned t0) {
  const unsigned dt = ticks() - t0;
  if (dt < 20000) return false;
  if (dt > DEADLINE || __builtin_amdgcn_readfirstlane(__hip_atomic_load(tmo, __ATOMIC_RELAXED, __HIP_MEMORY_SCOPE_AGENT)) == call) {
    __hip_atomic_store(tmo, call, __ATOMIC_RELAXED, __HIP_MEMORY_SCOPE_AGENT);
    asm volatile("s_waitcnt vmcnt(0)" ::: "memory");
    return true;
  }
  return false;
}
__device__ __forceinline__ unsigned sload(const void* p) { unsigned r; asm volatile("s_load_dword %0, %1, 0x0 glc\n\ts_waitcnt lgkmcnt(0)" : "=s"(r) : "s"(p) : "memory"); return r; }
__device__ __forceinline__ u32x8 sload8(const void* p) { u32x8 r; asm volatile("s_load_dwordx8 %0, %1, 0x0 glc\n\ts_waitcnt lgkmcnt(0)" : "=s"(r) : "s"(p) : "memory"); return r; }
#define MFMA(acc, a, b) asm volatile("v_mfma_f32_16x16x32_f16 %0, %1, %2, %0" : "+v"(acc) : "v"(a), "v"(b))

template <int STA>
__device__ __forceinline__ void ring_reduce(f32x4 (&acc)[2][8], float* out, int vo, int kq, __amdgpu_buffer_rsrc_t rc, int fo,
                                            unsigned call, GAS unsigned* tmo, unsigned t0) {
  auto ro = __builtin_amdgcn_make_buffer_rsrc((void*)out, 0, MT * ROW, 0x00020000);
#pragma unroll
  for (int v = 0; v < 4; v++) {
    const int i = v >> 1, c0 = (v & 1) * 4, blk = v ^ kq ^ 2;
    const int so = (blk >> 1) * 16 * ROW + (blk & 1) * 256;
    f32x4 o[4];
    if (v > 0) {
#pragma unroll
      for (int e = 0; e < 4; e++) { o[e] = __builtin_bit_cast(f32x4, __builtin_amdgcn_raw_buffer_load_b128(ro, vo, so + e * ROW, 16)); SB(); }
    }
    if (v == 3 && __hip_atomic_load(tmo, __ATOMIC_RELAXED, __HIP_MEMORY_SCOPE_AGENT) == call) {
      const float q = __builtin_nanf("");
#pragma unroll
      for (int e = 0; e < 4; e++) o[e] = f32x4{q, q, q, q};
    }
    const unsigned done = call * 4 + v + 1;
    const int fv = fo + blk * 64, fn = fo + ((v + 1) ^ kq ^ 2) * 64;
    unsigned dv = done;
    if (v < 3) asm volatile("" : "+v"(dv) : "s"(fv), "s"(fn));
#pragma unroll
    for (int e = 0; e < 4; e++) {
#pragma unroll
      for (int c = 0; c < 4; c++) {
        float t_;
        if (v > 0) { t_ = o[e][c]; asm volatile("v_add_f32 %0, %0, %1" : "+v"(t_) : "v"(acc[i][c0 + c][e])); }
        else t_ = acc[i][c0 + c][e];
        o[e][c] = t_;
      }
      asm volatile("" : "+v"(o[e]));
      __builtin_amdgcn_raw_buffer_store_b128(__builtin_bit_cast(u32x4, o[e]), ro, vo, so + e * ROW, STA);
      SB();
      asm volatile("s_nop 1");
      SB();
    }
    if (v < 3) {
      unsigned seen, n = 64, tv;
#define GLUE(SC) asm volatile("s_waitcnt vmcnt(0)\n\ts_mov_b64 exec, 1\n\tbuffer_store_dword %7, off, %3, %4" SC "\n" \
                   "1:\n\tbuffer_load_dword %2, off, %3, %5 sc1\n\ts_waitcnt vmcnt(0)\n\tv_readfirstlane_b32 %0, %2\n\t" \
                   "s_cmp_ge_u32 %0, %6\n\ts_cbranch_scc1 2f\n\ts_sub_u32 %1, %1, 1\n\ts_cbranch_scc0 1b\n2:\n\ts_mov_b64 exec, -1" \
                   : "=&s"(seen), "+s"(n), "=&v"(tv) : "s"(rc), "s"(fv), "s"(fn), "s"(done), "v"(dv) : "memory", "scc")
      if (STA) GLUE(" sc1"); else GLUE("");
      while (__builtin_expect(seen < done, 0)) {
        if (expired(tmo, call, t0)) break;
        __builtin_amdgcn_s_sleep(2);
        seen = __builtin_amdgcn_readfirstlane(__builtin_amdgcn_raw_buffer_load_b32(rc, 0, fn, 16));
        MEMBAR();
      }
      MEMBAR();
    }
    SB();
  }
}

__global__ __launch_bounds__(768, 3) void a2a_gemm(Params p) {
  extern __shared__ __attribute__((aligned(16))) char smem_[];
  LAS char* smem = (LAS char*)smem_;
  LAS unsigned* ctl = (LAS unsigned*)(smem + LDS_CTL);
  u32x8 ka; unsigned karank; unsigned long long tclk;
  asm volatile("s_load_dwordx8 %0, %3, 0x0\n\ts_load_dword %1, %3, 0x20\n\ts_memrealtime %2"
               : "=&s"(ka), "=&s"(karank), "=&s"(tclk) : "s"(__builtin_amdgcn_kernarg_segment_ptr()));
  const int lane = threadIdx.x & 63;
  const int wave = __builtin_amdgcn_readfirstlane(threadIdx.x >> 6);
  if (wave == 0) {
    lds_st(ctl + lane, 0); lds_st(ctl + C_ORD + lane, 32);
    asm volatile("s_waitcnt lgkmcnt(0)\n\ts_barrier" ::: "memory");
  } else if (wave < 8) {
    unsigned long long pc; unsigned junk;
    const unsigned off = min(((blockIdx.x >> 3) * 7 + wave - 1) * 64u, CPF - 64u);
    asm volatile("s_getpc_b64 %0\n\ts_load_dword %1, %0, %2\n\ts_barrier\n\ts_waitcnt lgkmcnt(0)" : "=&s"(pc), "=&s"(junk) : "s"(off) : "memory");
  }
  const int bid = blockIdx.x;
  const int idx = bid >> 3;
  const int kq = idx & 3, mh = (idx >> 2) & 1, ntl = idx >> 3, nt = (bid & 7) * 4 + ntl;
  const int un = (~ntl & 1) * 128 + ((mh ^ (ntl >> 1)) & 1) * 64 + (kq & 1) * 32 + (kq >> 1) * 16 + (ntl >> 1) * 8 + (bid & 7);
  const bool early = un < EARLY;
  int vw = ((lane >> 5) * 8 + (wave - 10) * 16 + kq * 32) * ROW + nt * 512 + (lane & 31) * 16;
  asm volatile("s_waitcnt lgkmcnt(0)" : "+s"(ka), "+s"(karank), "+s"(tclk), "+v"(vw));
  const float* px = (const float*)(((unsigned long long)ka[1] << 32) | ka[0]);
  const float* pw = (const float*)(((unsigned long long)ka[3] << 32) | ka[2]);
  float* pout = (float*)(((unsigned long long)ka[5] << 32) | ka[4]);
  char* myws = (char*)(((unsigned long long)ka[7] << 32) | ka[6]);
  const int rank = karank;
  const unsigned tstart = (unsigned)tclk;
  GAS unsigned* tmo = (GAS unsigned*)(myws + OFF_TMO);


  if (__builtin_expect(wave >= 10, 1)) {
    const int h = wave - 10;
    auto rw = __builtin_amdgcn_make_buffer_rsrc((void*)pw, 0, K * ROW, 0x00020000);
    if (early) __builtin_amdgcn_s_barrier();
    if (un < HOLD) lds_wait_ge(ctl + C_SENT, 1);
    MEMBAR();
    u32x4 wb[16];
#pragma unroll
    for (int n = 0; n < 16; n++) { wb[n] = __builtin_amdgcn_raw_buffer_load_b128(rw, vw, (rank * 4 + (n >> 3)) * (128 * ROW) + (n & 7) * ROW, 0); SB(); }
    if (!early) __builtin_amdgcn_s_barrier();
    MEMBAR();
    const int wl = (((lane >> 4) ^ kq) & 1) * 4096 + (h * 2 + (lane >> 5)) * 256 + (lane & 15) * 16;
    const int miss = 2 * ((kq + 1) & 3) + 1;
#pragma nounroll
    for (int q = 0; q < 32; q += 2) {
      const int ve = (q < 30) ? vw : 0x40000000;
#pragma unroll
      for (int d = 0; d < 2; d++) {
        const int t = q + d, tn = (t + 2) & 31;
        const int u = tn - 4, j7 = u - 7 * ((u * 37) >> 8), i8 = j7 + (j7 >= miss ? 1 : 0);
        int src; asm("s_cmp_lt_i32 %1, 4\n\ts_cselect_b32 %0, %2, %3" : "=s"(src) : "s"(tn), "s"(rank), "s"(rank + 1 + ((kq - (i8 >> 1)) & 3) + 4 * (i8 & 1)) : "scc");
        const int so = ((src & 7) * 4 + (tn & 3)) * (128 * ROW);
        u32x4 o[4];
#pragma unroll
        for (int i = 0; i < 4; i++) {
          const int m = d * 8 + 2 * i;
          const f32x4 w0 = __builtin_bit_cast(f32x4, wb[m]), w1 = __builtin_bit_cast(f32x4, wb[m + 1]);
#pragma unroll
          for (int j = 0; j < 4; j++) o[j][i] = cvtpk(w0[j], w1[j]);
          SB();
          wb[m] = __builtin_amdgcn_raw_buffer_load_b128(rw, ve, so + (2 * i) * ROW, 0); SB();
          wb[m + 1] = __builtin_amdgcn_raw_buffer_load_b128(rw, ve, so + (2 * i + 1) * ROW, 0); SB();
        }
        if (t >= RING) {
          for (;;) {
            const unsigned x = lds_ldv(ctl + ((lane & 8) ? C_INV + t - RING : C_DONE + (lane & 7)));
            const unsigned e = __builtin_amdgcn_readlane(x, 8);
            if (e != 0 && __ballot(x < e) == 0) break;
            __builtin_amdgcn_s_sleep(1);
          }
        }
        MEMBAR();
        LAS u32x4* wd = (LAS u32x4*)(smem + (t >= RING ? t - RING : t) * 8192 + wl);
#pragma unroll
        for (int j = 0; j < 4; j++) wd[j * 64] = o[j];
        MEMBAR();
        lds_st(ctl + h, t + 1);
        SB();
      }
    }
    {
      auto ro = __builtin_amdgcn_make_buffer_rsrc((void*)pout, 0, MT * ROW, 0x00020000);
      unsigned sink = 0;
#pragma unroll
      for (int q = 0; q < 2; q++)
        sink ^= __builtin_amdgcn_raw_buffer_load_b32(ro, (mh * 256 + kq * 64 + (h * 2 + q) * 16 + (lane >> 2)) * ROW + nt * 512 + (lane & 3) * 128, 0, 16);
      if (lane < 3) sink ^= __hip_atomic_load((GAS unsigned*)(myws + (lane == 2 ? OFF_TMO : OFF_CFLAG + (nt * 2 + mh) * 2048 + (kq * 4 + h * 2 + lane) * 128)), __ATOMIC_RELAXED, __HIP_MEMORY_SCOPE_AGENT);
      asm volatile("" :: "v"(sink));
    }
    return;
  }

  if (__builtin_expect(wave == 9, 1)) {
    const bool local = un < 32;
    const int sl = local ? un : ((un - 32) * 293) >> 11;
    const int dst = (local ? rank : rank + 1 + (un - 32) - 7 * sl) & 7;
    const unsigned tslot = local ? 0 : SLOT0 + ((unsigned)sl * SLOTLEN >> 3);
    const int r = (rank - dst - 1) & 7;
    const int gs = sl >> 3, gi = (sl >> 1) & 3, smh = sl & 1, skq = (gi + r) & 3;
    const int i8 = 2 * gi + (r >> 2);
    const int tpos = local ? gs : 4 + 7 * gs + i8 - (i8 > 2 * ((skq + 1) & 3) + 1 ? 1 : 0);
    const GAS float* xr = (const GAS float*)px + (size_t)(dst * MT + smh * 256 + (lane & 15)) * KS + (lane >> 4) * 8 + (tpos & 3) * 128 + skq * 32;
    __builtin_amdgcn_s_setprio(3);
    SB();
    if (!early) {
      __builtin_amdgcn_s_barrier();
      while (ticks() - tstart < XAT) __builtin_amdgcn_s_sleep(2);
    }
    MEMBAR();
    SB();
    f32x4 xv[32];
#pragma unroll
    for (int u = 0; u < 16; u++) {
      xv[u * 2] = *(const GAS f32x4*)(xr + u * 16 * KS);
      xv[u * 2 + 1] = *(const GAS f32x4*)(xr + u * 16 * KS + 4);
      SB();
    }
    if (early) __builtin_amdgcn_s_barrier();
    SB();
    const unsigned call = lds_wait_ge(ctl + C_CALL, 1);
    char* dbase = p.ws[dst];
    const unsigned a0 = OFF_A + (call & 1) * A_BYTES + tpos * 131072 + (smh * 4 + skq) * 16384 + lane * 16;
    const unsigned fo = OFF_AFLAG + ((smh * 4 + skq) * 4 + (tpos >> 3)) * 128 + (tpos & 7) * 4 + (lane & 3) * 4096;
    u32x4 o[16];
#pragma unroll
    for (int u = 0; u < 16; u++)
      o[u] = u32x4{cvtpk(xv[2 * u][0], xv[2 * u][1]), cvtpk(xv[2 * u][2], xv[2 * u][3]), cvtpk(xv[2 * u + 1][0], xv[2 * u + 1][1]), cvtpk(xv[2 * u + 1][2], xv[2 * u + 1][3])};
    while (ticks() - tstart < tslot) __builtin_amdgcn_s_sleep(1);
#pragma unroll
    for (int u = 0; u < 16; u++)
      asm volatile("global_store_dwordx4 %0, %1, %2 offset:%3 sc0 sc1\n\ts_nop 1" :: "v"(a0 + (u >> 2) * 4096), "v"(o[u]), "s"(dbase), "i"((u & 3) * 1024) : "memory");
    asm volatile("s_waitcnt vmcnt(0)" ::: "memory");
    asm volatile("s_mov_b64 exec, 15\n\tglobal_store_dword %0, %1, %2 sc0 sc1\n\ts_mov_b64 exec, -1" :: "v"(fo), "v"(call), "s"(dbase) : "memory");
    lds_st(ctl + C_SENT, 1);
    GAS unsigned* xt = (GAS unsigned*)(myws + OFF_XTAB + (nt * 2 + mh) * 64);
    unsigned xe;
    for (;;) {
      xe = __hip_atomic_load(xt + (lane & 3), __ATOMIC_RELAXED, __HIP_MEMORY_SCOPE_AGENT);
      if (__ballot((xe >> 4) != call) == 0 || expired(tmo, call, tstart)) break;
      __builtin_amdgcn_s_sleep(8);
    }
    lds_st(ctl + C_MODE, __ballot(xe != (unsigned)__builtin_amdgcn_readfirstlane(xe)) == 0 ? 1 : 2);
    return;
  }

  if (wave == 8) {
    __builtin_amdgcn_s_setprio(3);
    __builtin_amdgcn_s_barrier();
    const unsigned call = sload(myws + OFF_CNT + bid * 128) + 1;
    lds_st(ctl + C_CALL, call);
    const unsigned xcc = __builtin_amdgcn_s_getreg((3 << 11) | 20);
    GAS unsigned* xt = (GAS unsigned*)(myws + OFF_XTAB + (nt * 2 + mh) * 64);
    if (lane == 0) {
      __hip_atomic_store((GAS unsigned*)(myws + OFF_CNT + bid * 128), call, __ATOMIC_RELAXED, __HIP_MEMORY_SCOPE_AGENT);
      __hip_atomic_store(xt + kq, call * 16 + xcc, __ATOMIC_RELAXED, __HIP_MEMORY_SCOPE_AGENT);
    }
    const char* af = myws + OFF_AFLAG + (bid & 3) * 4096 + (mh * 4 + kq) * 512;
    while (ticks() - tstart < POLL0) __builtin_amdgcn_s_sleep(2);
    int got = 0;
    unsigned todo = 0xffffffffu, here = 0;
    auto look = [&](int line) {
      const u32x8 f = sload8(af + line * 128);
      unsigned m = 0;
#pragma unroll
      for (int q = 0; q < 8; q++) m |= (f[q] >= call ? 1u : 0u) << q;
      here |= m << (line * 8);
    };
    while (todo) {
      const int low = __builtin_ctz(todo);
      look(low >> 3);
      const unsigned missing = todo & ~here;
      unsigned take = missing ? todo & ((1u << __builtin_ctz(missing)) - 1) : todo;
      if (!take && low >= 4) {
        const unsigned x = lds_ldv(ctl + ((lane & 8) ? (lane & 1) : C_DONE + (lane & 7)));
        if (__ballot((lane & 8) == 0 && (int)x + DYN > got) != 0) {
          const int lim = min((int)__builtin_amdgcn_readlane(x, 8), (int)__builtin_amdgcn_readlane(x, 9));
          unsigned cand = todo & here;
          if (!cand && low < 24) { look((low >> 3) + 1); cand = todo & here; }
          if (!cand && low < 16) { look((low >> 3) + 2); cand = todo & here; }
          if (cand && __builtin_ctz(cand) < lim) take = cand & -cand;
        }
      }
      if (take) {
        const int e = got + __builtin_popcount(take & ((1u << (lane & 31)) - 1));
        if (lane < 32 && ((take >> lane) & 1)) { lds_st(ctl + C_ORD + e, lane); lds_st(ctl + C_INV + lane, e + 1); }
        MEMBAR();
        const int n = got + __builtin_popcount(take);
        got = n; todo &= ~take;
        lds_st(ctl + C_AST, n == 32 ? 64 : n);
        if (todo && (__builtin_ctz(todo) >> 3) != (low >> 3)) continue;
      } else if (ticks() - tstart > 5000) {
        const unsigned f = __hip_atomic_load((const GAS unsigned*)(af + ((lane >> 3) & 3) * 128) + (lane & 7), __ATOMIC_RELAXED, __HIP_MEMORY_SCOPE_SYSTEM);
        here |= (unsigned)__ballot(f >= call);
        if (expired(tmo, call, tstart)) here = 0xffffffffu;
      }
      __builtin_amdgcn_s_sleep(PSLEEP);
    }
    return;
  }

  const unsigned call = lds_wait_ge(ctl + C_CALL, 1);
  auto ra = __builtin_amdgcn_make_buffer_rsrc((void*)(myws + OFF_A + (call & 1) * A_BYTES + (mh * 4 + kq) * 16384), 0, 31 * 131072 + 16384, 0x00020000);
  const int vl = lane * 16;
  const int va0 = vl + wave * 2048 + ((1 + (kq >> 1)) & 1) * 1024, va1 = va0 ^ 1024;
  f32x4 acc[2][8];
#pragma unroll
  for (int i = 0; i < 2; i++)
#pragma unroll
    for (int q = 0; q < 8; q++) { acc[i][q] = f32x4{0, 0, 0, 0}; asm volatile("" : "+v"(acc[i][q])); }
  u32x4 ab[4][2], bf[8];
  int wlim = 0;
  int alim = 0;
  int ord = 0;
  auto wgate = [&](int t) {
    if (t >= wlim) {
      do {
        const int w = min(lds_ld(ctl + C_W0), lds_ld(ctl + C_W1));
        wlim = (w >= 32) ? 64 : w;
        if (t >= wlim) __builtin_amdgcn_s_sleep(1);
      } while (t >= wlim);
    }
    MEMBAR();
  };
  auto agate = [&](int e) {
    if (e >= alim) { alim = lds_wait_ge(ctl + C_AST, e + 1); ord = lds_ldv(ctl + C_ORD + lane); asm volatile("" : "+v"(ord)); }
    MEMBAR();
    return __builtin_amdgcn_readlane(ord, e);
  };
  wgate(0);
#pragma unroll
  for (int q = 0; q < 8; q++) { bf[q] = *(LAS const u32x4*)(smem + vl + q * 1024); SB(); }
  agate(1);
#pragma unroll
  for (int d = 0; d < 2; d++) {
    ab[d][0] = __builtin_amdgcn_raw_buffer_load_b128(ra, va0, d * 131072, 17); SB();
    ab[d][1] = __builtin_amdgcn_raw_buffer_load_b128(ra, va1, d * 131072, 17); SB();
  }
  unsigned mode = 0;
#pragma nounroll
  for (int t0 = 0; t0 < 32; t0 += 4) {
#pragma unroll
    for (int u = 0; u < 4; u++) {
      const int t = t0 + u;
      if (u == 2 && t0 == 28) mode = lds_ld(ctl + C_MODE);
      const int pa = agate(t + 2);
      SB();
      ab[(u + 2) & 3][0] = __builtin_amdgcn_raw_buffer_load_b128(ra, va0, pa * 131072, 17); SB();
      ab[(u + 2) & 3][1] = __builtin_amdgcn_raw_buffer_load_b128(ra, va1, pa * 131072, 17); SB();
      const int pn = __builtin_amdgcn_readlane(ord, t + 1);
      wgate(pn);
      LAS const u32x4* wp = (LAS const u32x4*)(smem + vl + (pn >= RING ? pn - RING : pn) * 8192);
      SB();
#pragma unroll
      for (int q = 0; q < 8; q++) {
        MFMA(acc[0][q], ab[u][0], bf[q]);
        MFMA(acc[1][q], ab[u][1], bf[q]);
        SB();
        bf[q] = wp[q * 64];
        SB();
      }
      MEMBAR();
      lds_st(ctl + C_DONE + wave, t + 2);
      SB();
    }
  }
  if (mode == 0) mode = lds_wait_ge(ctl + C_MODE, 1);
  auto rc = __builtin_amdgcn_make_buffer_rsrc((void*)(myws + OFF_CFLAG), 0, 131072, 0x00020000);
  const int vo = (mh * 256 + wave * 32 + (lane >> 4) * 4) * ROW + nt * 512 + (lane & 15) * 16;
  const int fo = ((nt * 2 + mh) * 8 + wave) * 256;
  asm volatile("s_nop 7"
               : "+v"(acc[0][0]), "+v"(acc[0][1]), "+v"(acc[0][2]), "+v"(acc[0][3]), "+v"(acc[0][4]), "+v"(acc[0][5]), "+v"(acc[0][6]), "+v"(acc[0][7]),
                 "+v"(acc[1][0]), "+v"(acc[1][1]), "+v"(acc[1][2]), "+v"(acc[1][3]), "+v"(acc[1][4]), "+v"(acc[1][5]), "+v"(acc[1][6]), "+v"(acc[1][7]));
  SB();
  if (__builtin_expect(mode == 1, 1)) ring_reduce<0>(acc, pout, vo, kq, rc, fo, call, tmo, tstart);
  else ring_reduce<16>(acc, pout, vo, kq, rc, fo, call, tmo, tstart);
}

extern "C" void kernel_launch(int rank, int nranks, ncclComm_t comm, hipStream_t stream,
                              void* const* d_in, const int* in_sizes, int n_in,
                              void* d_out, int out_size, const int* in_dtypes, int out_dtype,
                              void* const* d_out_all, void* const* d_ws_all, size_t ws_bytes) {
  if (nranks != NR || (unsigned)rank >= (unsigned)NR || ws_bytes < WS_NEED || n_in != 2 || in_sizes[0] != NR * MT * KS ||
      in_sizes[1] != K * N || out_size != MT * N) { fprintf(stderr, "unexpected configuration\n"); return; }
  static bool init = false;
  if (!init) { HIP_CHECK(hipFuncSetAttribute((const void*)a2a_gemm, hipFuncAttributeMaxDynamicSharedMemorySize, LDS_BYTES)); init = true; }
  Params p;
  p.x = (const float*)d_in[0]; p.w = (const float*)d_in[1]; p.out = (float*)d_out;
  p.myws = (char*)d_ws_all[rank]; p.rank = rank; p.pad = 0;
  for (int i = 0; i < NR; i++) p.ws[i] = (char*)d_ws_all[i];
  hipLaunchKernelGGL(a2a_gemm, dim3(256), dim3(768), LDS_BYTES, stream, p);
  HIP_CHECK(hipGetLastError());
}
```
